```python
import numpy as np

kernel_name = "gpu_gemm_a2a_m4096_k4096_n16384_bf16_silu"


ABSMAX_RTOL = 0.008

COMPUTE_DTYPE = "bfloat16"
EPILOGUE = "silu"
IN_QUANT = None
OUT_QUANT = None
IN_DTYPES = {"x": "bfloat16", "w_mat": "bfloat16"}
WORLD_SIZE = 8

def _to_bf16(x):
    u = np.ascontiguousarray(x, dtype=np.float32).view(np.uint32)
    return ((u + 0x7FFF + ((u >> 16) & 1)) & 0xFFFF0000).view(np.float32)

def setup_inputs():

    x = _to_bf16(np.random.randn(4096, 4096).astype(np.float32))
    w_mat = _to_bf16((np.random.randn(4096, 16384) / np.sqrt(4096)).astype(np.float32))
    return {"x": x, "w_mat": w_mat}

def _epilogue(y):
    return (y / (1.0 + np.exp(-np.clip(y, -60.0, 60.0)))).astype(np.float32)

def reference(x, w_mat):
    y = (x @ w_mat).astype(np.float32)
    return _epilogue(y)
```

```cpp
#include <hip/hip_runtime.h>
#include <rccl/rccl.h>
#include <cstdio>

#define HIP_CHECK(expr) do { hipError_t _e = (expr); if (_e != hipSuccess) { \
    fprintf(stderr, "HIP error %s at %s:%d\n", hipGetErrorString(_e), __FILE__, __LINE__); \
    return; } } while (0)

#define LAS __attribute__((address_space(3)))
typedef short s16x4 __attribute__((ext_vector_type(4)));
typedef short s16x8 __attribute__((ext_vector_type(8)));
typedef float f32x4 __attribute__((ext_vector_type(4)));
typedef unsigned u32x4 __attribute__((ext_vector_type(4)));
typedef __bf16 bf16x8 __attribute__((ext_vector_type(8)));

namespace {

constexpr int NRANK = 8;
constexpr int MLOC = 512, KDIM = 4096, NDIM = 16384, NOUT = 2048, NKT = KDIM / 64;
constexpr int BM = 128, BN = 128, MF = 4, NF = 4, NACC = 16, NS = 4;
constexpr int ABYTES = BM * 128, BBYTES = BN * 128, STAGE = ABYTES + BBYTES;
constexpr int XOFF = NS * STAGE;
constexpr int LDS_BYTES = XOFF + 32768;
constexpr int NLD = 16;
constexpr int NWG = 256;
constexpr int NCHUNK = 4;
constexpr int GROWS = 256;
constexpr size_t ROWB = (size_t)KDIM * 2;
constexpr size_t WS_CALL = 0, WS_TMO = 128, WS_GCNT = 1024, WS_OCNT = 8192, WS_G = 1u << 20;
constexpr size_t G_SRC = (size_t)GROWS * ROWB;
constexpr size_t G_PAR = NRANK * G_SRC;
constexpr size_t WS_NEED = WS_G + 2 * G_PAR;
constexpr unsigned long long DEADLINE_TICKS = 50000000ull;
constexpr int PACE_NUM = 3, PACE_DEN = 2, LAG = 6;
constexpr int NGJOB = 8 * 7;
constexpr int DUMP2_T = 46;

struct Params { const char* X; const char* W; char* out[NRANK]; char* ws[NRANK]; int rank; };

template <int N> __device__ __forceinline__ void wait_vm() { asm volatile("s_waitcnt vmcnt(%0)" :: "n"(N) : "memory"); }
__device__ __forceinline__ void wait_lgkm0() { asm volatile("s_waitcnt lgkmcnt(0)" ::: "memory"); }
__device__ __forceinline__ unsigned cvt_pk_bf16(float lo, float hi) { unsigned r; asm("v_cvt_pk_bf16_f32 %0, %1, %2" : "=v"(r) : "v"(lo), "v"(hi)); return r; }
__device__ __forceinline__ float silu(float y) {
    float c = __builtin_amdgcn_fmed3f(y, -60.f, 60.f);
    float e = __builtin_amdgcn_exp2f(c * -1.4426950408889634f);
    return y * __builtin_amdgcn_rcpf(1.0f + e);
}
__device__ __forceinline__ void glds(unsigned voff, const char* sbase, unsigned ldsdst) {
    asm volatile("s_mov_b32 m0, %2\n\ts_nop 0\n\tglobal_load_lds_dwordx4 %0, %1" :: "v"(voff), "s"(sbase), "s"(ldsdst) : "memory");
}
__device__ __forceinline__ void store_wt(char* p, u32x4 v) {
    asm volatile("global_store_dwordx4 %0, %1, off sc0 sc1\n\ts_nop 1" :: "v"(p), "v"(v) : "memory");
}
__device__ __forceinline__ u32x4 finish_pair(f32x4 v0, f32x4 v1) {
    unsigned ax = cvt_pk_bf16(silu(v0[0]), silu(v0[1])), ay = cvt_pk_bf16(silu(v0[2]), silu(v0[3]));
    unsigned bx = cvt_pk_bf16(silu(v1[0]), silu(v1[1])), by = cvt_pk_bf16(silu(v1[2]), silu(v1[3]));
    auto rx = __builtin_amdgcn_permlane16_swap(ax, bx, false, false);
    auto ry = __builtin_amdgcn_permlane16_swap(ay, by, false, false);
    return (u32x4){rx[0], ry[0], rx[1], ry[1]};
}
#define BAR __builtin_amdgcn_s_barrier()
#define SCHED __builtin_amdgcn_sched_barrier(0)

__device__ __forceinline__ void spin_ge(const unsigned* flag, unsigned want, unsigned* tmo, unsigned call) {
    const unsigned long long t0 = __builtin_amdgcn_s_memrealtime();
    while ((unsigned)__builtin_amdgcn_readfirstlane(__hip_atomic_load(flag, __ATOMIC_RELAXED, __HIP_MEMORY_SCOPE_SYSTEM)) < want) {
        if ((unsigned)__builtin_amdgcn_readfirstlane(__hip_atomic_load(tmo, __ATOMIC_RELAXED, __HIP_MEMORY_SCOPE_AGENT)) == call
            || __builtin_amdgcn_s_memrealtime() - t0 > DEADLINE_TICKS) {
            __hip_atomic_store(tmo, call, __ATOMIC_RELAXED, __HIP_MEMORY_SCOPE_AGENT);
            break;
        }
        __builtin_amdgcn_s_sleep(2);
    }
}

__global__ __launch_bounds__(512, 2) void gemm_a2a_kernel(Params p) {
    extern __shared__ __attribute__((aligned(16))) char smem_[];
    LAS char* smem = (LAS char*)smem_;
    const int tid = threadIdx.x, wid = __builtin_amdgcn_readfirstlane(tid >> 6), lane = tid & 63;
    const int fr = lane & 15, fq = lane >> 4;
    const int rank = p.rank;
    const int xcd = blockIdx.x & 7, c = blockIdx.x >> 3;
    const int rb0 = c & 1, slot0 = c >> 1;
    const int dest0 = (rank + 1 + (slot0 & 7)) & 7;
    const int col0 = (2 * xcd + (slot0 >> 3)) * BN;
    const int rb1 = 4 * (xcd & 3) + (c & 3), cb1 = 8 * (xcd >> 2) + (c >> 2);
    const int src1 = rb1 >> 1, h1 = rb1 & 1;
    char* const myws = p.ws[rank];
    LAS unsigned* callw = (LAS unsigned*)(smem + XOFF);

    if (wid < 4) {
        const int wr = wid >> 1, wc = wid & 1;
        const unsigned aoff0 = (unsigned)((wr * 64 + fr) * 128 + ((fq ^ (fr & 7)) * 16));
        const unsigned aoff1 = aoff0 ^ 64u;
        unsigned boff[2][2];
        { const int g = fq, q = fr >> 2, pp = fr & 3;
#pragma unroll
          for (int c1 = 0; c1 < 2; ++c1)
#pragma unroll
            for (int t = 0; t < 2; ++t)
              boff[c1][t] = (unsigned)(ABYTES + BN * 16 * g + 1024 * wc + 64 * (4 * t + q) + 16 * ((2 * c1 + (pp >> 1)) ^ ((2 * g + t) & 3)) + 8 * (pp & 1)); }
        f32x4 acc[NACC], oldv[NACC / 2];
#pragma unroll
        for (int i = 0; i < NACC; ++i) acc[i] = (f32x4){0.f, 0.f, 0.f, 0.f};
        s16x8 aA[MF], bA[NF], aB[MF], bB[NF];
        LAS char* xreg = smem + XOFF + wid * 8192 + lane * 16;
#define RD(a_, b_, stg, ks) do { LAS char* sb_ = smem + (stg) * STAGE; \
      _Pragma("unroll") for (int nf = 0; nf < NF; ++nf) { \
        s16x4 lo = __builtin_amdgcn_ds_read_tr16_b64_v4i16((LAS s16x4*)(sb_ + boff[nf & 1][0] + (ks) * (BN * 64) + (nf >> 1) * 512)); \
        s16x4 hi = __builtin_amdgcn_ds_read_tr16_b64_v4i16((LAS s16x4*)(sb_ + boff[nf & 1][1] + (ks) * (BN * 64) + (nf >> 1) * 512)); \
        b_[nf] = __builtin_shufflevector(lo, hi, 0, 1, 2, 3, 4, 5, 6, 7); } \
      _Pragma("unroll") for (int mf = 0; mf < MF; ++mf) a_[mf] = *(LAS s16x8*)(sb_ + ((ks) ? aoff1 : aoff0) + mf * 2048); } while (0)
#define MM(a_, b_) do { _Pragma("unroll") for (int mf = 0; mf < MF; ++mf) _Pragma("unroll") for (int nf = 0; nf < NF; ++nf) \
        acc[mf * NF + nf] = __builtin_amdgcn_mfma_f32_16x16x32_bf16(__builtin_bit_cast(bf16x8, b_[nf]), __builtin_bit_cast(bf16x8, a_[mf]), acc[mf * NF + nf], 0, 0, 0); } while (0)
#define ILV() do { _Pragma("unroll") for (int i_ = 0; i_ < MF + 2 * NF; ++i_) { __builtin_amdgcn_sched_group_barrier(0x008, 1, 0); __builtin_amdgcn_sched_group_barrier(0x100, 1, 0); } \
      __builtin_amdgcn_sched_group_barrier(0x008, NACC - (MF + 2 * NF), 0); } while (0)
#define KTILE() do { SCHED; RD(aB, bB, st, 1); MM(aA, bA); ILV(); SCHED; wait_lgkm0(); BAR; SCHED; \
      st = (st + 1) & (NS - 1); RD(aA, bA, st, 0); MM(aB, bB); ILV(); SCHED; } while (0)
        BAR;
        const unsigned call = (unsigned)__builtin_amdgcn_readfirstlane(*callw);
        int st = 0;
        RD(aA, bA, 0, 0);
        for (int t = 0; t < NKT; ++t) KTILE();
#pragma unroll
        for (int i = 0; i < NACC / 2; ++i) { *(LAS f32x4*)(xreg + i * 1024) = acc[i]; oldv[i] = acc[i + NACC / 2]; }
#pragma unroll
        for (int i = 0; i < NACC; ++i) acc[i] = (f32x4){0.f, 0.f, 0.f, 0.f};
        for (int t = 0; t <= DUMP2_T; ++t) KTILE();
#pragma unroll
        for (int i = 0; i < NACC / 2; ++i) *(LAS f32x4*)(xreg + i * 1024) = oldv[i];
        for (int t = DUMP2_T + 1; t < NKT - 4; ++t) KTILE();
        const unsigned tm = __hip_atomic_load((unsigned*)(myws + WS_TMO), __ATOMIC_RELAXED, __HIP_MEMORY_SCOPE_AGENT);
        for (int t = NKT - 4; t < NKT; ++t) KTILE();
        const bool bad = tm == call;
        char* optr = p.out[rank] + ((size_t)(src1 * MLOC + 256 + h1 * BM + wr * 64 + fr) * NOUT + cb1 * BN + wc * 64 + (fq & 1) * 16 + (fq >> 1) * 8) * 2;
#pragma unroll
        for (int i = 0; i < NACC / 2; ++i) {
            u32x4 wv = finish_pair(acc[2 * i], acc[2 * i + 1]);
            if (bad) wv = (u32x4){0x7fc07fc0u, 0x7fc07fc0u, 0x7fc07fc0u, 0x7fc07fc0u};
            *(u32x4*)(optr + (size_t)(i >> 1) * 16 * NOUT * 2 + (i & 1) * 64) = wv;
        }
    } else if (wid < 6) {
        const int lw = wid - 4;
        unsigned voffA[8], voffB[4][2];
#pragma unroll
        for (int j = 0; j < 8; ++j) voffA[j] = (unsigned)((lw * 64 + j * 8 + (lane >> 3)) * (KDIM * 2) + (((lane & 7) ^ (lane >> 3)) * 16));
#pragma unroll
        for (int e = 0; e < 4; ++e)
#pragma unroll
            for (int cb = 0; cb < 2; ++cb) { const int r = (lane >> 2) & 7, j = lane >> 5, pos = lane & 3;
                voffB[e][cb] = (unsigned)((8 * (4 * lw + e) + r) * (NDIM * 2) + (4 * j + (pos ^ ((2 * e + (r >> 2)) & 3))) * 16 + cb * 128); }
        const char* A0 = p.X + (size_t)rb0 * BM * ROWB;
        const char* B0 = p.W + (size_t)(dest0 * NOUT + col0) * 2;
        const char* B1 = p.W + (size_t)(rank * NOUT + cb1 * BN) * 2;
        const unsigned lds0 = (unsigned)(size_t)smem;
        const unsigned ldA_dst = lds0 + lw * 8192, ldB_dst = lds0 + ABYTES + BN * 64 * lw;
        int ld_stage = 0;
#define LOADT(Ap, Bp, kt) do { const char* sA = (Ap) + (size_t)(kt) * 128; const char* sB = (Bp) + (size_t)(kt) * (64 * NDIM * 2); \
      const unsigned dA = ldA_dst + ld_stage * STAGE, dB = ldB_dst + ld_stage * STAGE; ld_stage = (ld_stage + 1) & (NS - 1); \
      _Pragma("unroll") for (int e = 0; e < 4; ++e) _Pragma("unroll") for (int cb = 0; cb < 2; ++cb) glds(voffB[e][cb], sB, dB + e * BN * 16 + cb * 1024); \
      _Pragma("unroll") for (int j = 0; j < 8; ++j) glds(voffA[j], sA, dA + j * 1024); } while (0)
#pragma unroll
        for (int t = 0; t < NS; ++t) LOADT(A0, B0, t);
        wait_vm<(NS - 1) * NLD>(); BAR;
        const unsigned call = (unsigned)__builtin_amdgcn_readfirstlane(*callw);
        const char* A1 = (src1 == rank) ? p.X + (size_t)(256 + h1 * BM) * ROWB
                                        : myws + WS_G + (size_t)(call & 1) * G_PAR + (size_t)src1 * G_SRC + (size_t)h1 * BM * ROWB;
        for (int t = 0; t < NKT - NS; ++t) { wait_vm<(NS - 2) * NLD>(); BAR; LOADT(A0, B0, t + NS); }
        for (int t = 0; t < NKT; ++t) { wait_vm<(NS - 2) * NLD>(); BAR; LOADT(A1, B1, t); }
        for (int t = 0; t < NS; ++t) { wait_vm<(NS - 2) * NLD>(); BAR; LOADT(A1, B1, t); }
        wait_vm<0>();
    } else if (wid == 6) {
        u32x4 xr[8];
        const char* xsrc = p.X + (size_t)(256 + blockIdx.x) * ROWB + lane * 16;
#pragma unroll
        for (int i = 0; i < 8; ++i) xr[i] = *(const u32x4*)(xsrc + i * 1024);
        const unsigned pl = (unsigned)(rank + 1 + (lane < 7 ? lane : 0)) & 7u;
        char* peer_ws_l = p.ws[0];
#pragma unroll
        for (int q = 1; q < NRANK; ++q) peer_ws_l = (pl == (unsigned)q) ? p.ws[q] : peer_ws_l;
        unsigned* gcnt_l = (unsigned*)(peer_ws_l + WS_GCNT + (size_t)rank * NCHUNK * 128);
        asm volatile("" : "+v"(gcnt_l));
        unsigned tk = 0;
        if (lane == 0) tk = __hip_atomic_fetch_add((unsigned*)(myws + WS_CALL), 1u, __ATOMIC_RELAXED, __HIP_MEMORY_SCOPE_AGENT);
        const unsigned call = (unsigned)__builtin_amdgcn_readfirstlane(tk) / NWG + 1u;
        if (lane == 0) *callw = call;
        wait_vm<0>(); wait_lgkm0();
#pragma unroll
        for (int i = 0; i < 8; ++i) asm volatile("" : "+v"(xr[i]));
        BAR;
        int n = 0;
        int job = 0;
#define PACE(minn) do { int tgt_ = (job * PACE_NUM) / PACE_DEN; if (tgt_ < (minn)) tgt_ = (minn); if (tgt_ > 2 * NKT) tgt_ = 2 * NKT; while (n < tgt_) { BAR; ++n; } ++job; } while (0)
        const size_t goff = WS_G + (size_t)(call & 1) * G_PAR + (size_t)rank * G_SRC + (size_t)blockIdx.x * ROWB + lane * 16;
#define PUBLISH_G(ch) do { wait_vm<LAG>(); if (lane < 7) { unsigned* a_ = gcnt_l + (ch) * 32; asm volatile("global_atomic_add %0, %1, off sc1" :: "v"(a_), "v"(1u) : "memory"); } } while (0)
#pragma unroll
        for (int i = 0; i < 8; ++i) {
            for (int q = 0; q < 7; ++q) {
                PACE(0);
                if (i >= 2 && (i & 1) == 0 && q == LAG) PUBLISH_G(i / 2 - 1);
                store_wt(p.ws[(rank + 1 + q) & 7] + goff + i * 1024, xr[i]);
            }
        }
        char* optr = p.out[dest0] + ((size_t)(rank * MLOC + rb0 * BM + fr) * NOUT + col0 + (fq & 1) * 16 + (fq >> 1) * 8) * 2;
        LAS char* xd = smem + XOFF + lane * 16;
        for (int hf = 0; hf < 2; ++hf) {
            for (int j = 0; j < 16; ++j) {
                PACE(hf == 0 ? NKT + 1 : NKT + DUMP2_T + 2);
                if (hf == 0 && j == LAG) PUBLISH_G(NCHUNK - 1);
                const int w = j >> 2, i = j & 3;
                f32x4 v0 = *(LAS f32x4*)(xd + w * 8192 + (2 * i) * 1024), v1 = *(LAS f32x4*)(xd + w * 8192 + (2 * i + 1) * 1024);
                u32x4 wv = finish_pair(v0, v1);
                const int fi = hf * 4 + i;
                store_wt(optr + (size_t)((w >> 1) * 64 + (fi >> 1) * 16) * NOUT * 2 + (w & 1) * 128 + (fi & 1) * 64, wv);
            }
        }
        while (n < 2 * NKT) { BAR; ++n; }
        wait_vm<0>();
        if (dest0 != rank && lane == 0)
            __hip_atomic_fetch_add((unsigned*)(p.ws[dest0] + WS_OCNT + (size_t)rank * 128), 1u, __ATOMIC_RELAXED, __HIP_MEMORY_SCOPE_SYSTEM);
    } else {
        BAR;
        const unsigned call = (unsigned)__builtin_amdgcn_readfirstlane(*callw);
        unsigned* tmo = (unsigned*)(myws + WS_TMO);
        const unsigned* gc = (const unsigned*)(myws + WS_GCNT + (size_t)src1 * NCHUNK * 128);
        int n = 0;
        for (int ch = 0; ch < NCHUNK; ++ch) {
            const int need = NKT - NS + 16 * ch;
            while (n < need - 9) { BAR; ++n; }
            if (src1 != rank) {
                unsigned v = __hip_atomic_load(gc + ch * 32, __ATOMIC_RELAXED, __HIP_MEMORY_SCOPE_SYSTEM);
                BAR; BAR; BAR; BAR; n += 4;
                if ((unsigned)__builtin_amdgcn_readfirstlane(v) < NWG * call) spin_ge(gc + ch * 32, NWG * call, tmo, call);
                asm volatile("buffer_inv sc0 sc1" ::: "memory");
                BAR; BAR; BAR; BAR; n += 4;
                wait_vm<0>();
            }
        }
        while (n < 2 * NKT) { BAR; ++n; }
        if (blockIdx.x < NRANK && (int)blockIdx.x != rank) {
            spin_ge((const unsigned*)(myws + WS_OCNT + (size_t)blockIdx.x * 128), 32u * call, tmo, call);
            __builtin_amdgcn_fence(__ATOMIC_ACQUIRE, "");
            if (__hip_atomic_load(tmo, __ATOMIC_RELAXED, __HIP_MEMORY_SCOPE_AGENT) == call && lane == 0)
                *(unsigned short*)(p.out[rank] + (size_t)(rank * MLOC) * NOUT * 2) = 0x7fc0;
        }
    }
}

}

extern "C" void kernel_launch(int rank, int nranks, ncclComm_t comm,
                              hipStream_t stream,
                              void* const* d_in, const int* in_sizes, int n_in,
                              void* d_out, int out_size,
                              const int* in_dtypes, int out_dtype,
                              void* const* d_out_all,
                              void* const* d_ws_all, size_t ws_bytes) {
    (void)comm; (void)in_sizes; (void)n_in; (void)d_out; (void)out_size; (void)in_dtypes; (void)out_dtype;
    if (nranks != NRANK || ws_bytes < WS_NEED) { fprintf(stderr, "unsupported configuration\n"); return; }
    HIP_CHECK(hipFuncSetAttribute((const void*)gemm_a2a_kernel, hipFuncAttributeMaxDynamicSharedMemorySize, LDS_BYTES));
    Params p;
    p.X = (const char*)d_in[0];
    p.W = (const char*)d_in[1];
    for (int i = 0; i < NRANK; ++i) { p.out[i] = (char*)d_out_all[i]; p.ws[i] = (char*)d_ws_all[i]; }
    p.rank = rank;
    gemm_a2a_kernel<<<NWG, 512, LDS_BYTES, stream>>>(p);
    HIP_CHECK(hipGetLastError());
}
```

```cpp
#include <hip/hip_runtime.h>
#include <rccl/rccl.h>
#include <cstdio>

#define HIP_CHECK(expr) do { hipError_t _e = (expr); if (_e != hipSuccess) { \
    fprintf(stderr, "HIP error %s at %s:%d\n", hipGetErrorString(_e), __FILE__, __LINE__); \
    return; } } while (0)

#define LAS __attribute__((address_space(3)))
typedef short s16x4 __attribute__((ext_vector_type(4)));
typedef short s16x8 __attribute__((ext_vector_type(8)));
typedef float f32x4 __attribute__((ext_vector_type(4)));
typedef unsigned u32x4 __attribute__((ext_vector_type(4)));
typedef __bf16 bf16x8 __attribute__((ext_vector_type(8)));

namespace {

constexpr int NRANK = 8;
constexpr int MLOC = 512, KDIM = 4096, NDIM = 16384, NOUT = 2048, NKT = KDIM / 64;
constexpr int BM = 128, BN = 128, MF = 4, NF = 4, NACC = 16, NS = 4;
constexpr int ABYTES = BM * 128, BBYTES = BN * 128, STAGE = ABYTES + BBYTES;
constexpr int XOFF = NS * STAGE;
constexpr int LDS_BYTES = XOFF + 32768;
constexpr int NLD = 16;
constexpr int NWG = 256;
constexpr int NCHUNK = 4;
constexpr int GROWS = 256;
constexpr size_t ROWB = (size_t)KDIM * 2;
constexpr size_t WS_CALL = 0, WS_TMO = 128, WS_DONE = 256, WS_GCNT = 1024, WS_OCNT = 8192, WS_G = 1u << 20;
constexpr size_t G_SRC = (size_t)GROWS * ROWB;
constexpr size_t G_PAR = NRANK * G_SRC;
constexpr size_t WS_NEED = WS_G + 2 * G_PAR;
constexpr unsigned long long DEADLINE_TICKS = 50000000ull;
#define GSTEP_ 10
#define OSTEP_ 11
constexpr int GSTEP = GSTEP_, OSTEP = OSTEP_;
constexpr int DUMP2_T = 8 * GSTEP + GSTEP + 8 - NKT;

struct Params { const char* X; const char* W; char* out[NRANK]; char* ws[NRANK]; char* myws; char* myout; int rank; };

template <int N> __device__ __forceinline__ void wait_vm() { asm volatile("s_waitcnt vmcnt(%0)" :: "n"(N) : "memory"); }
__device__ __forceinline__ void wait_lgkm0() { asm volatile("s_waitcnt lgkmcnt(0)" ::: "memory"); }
__device__ __forceinline__ unsigned cvt_pk_bf16(float lo, float hi) { unsigned r; asm("v_cvt_pk_bf16_f32 %0, %1, %2" : "=v"(r) : "v"(lo), "v"(hi)); return r; }
__device__ __forceinline__ float silu(float y) {
    float c = __builtin_amdgcn_fmed3f(y, -60.f, 60.f);
    float e = __builtin_amdgcn_exp2f(c * -1.4426950408889634f);
    return y * __builtin_amdgcn_rcpf(1.0f + e);
}
__device__ __forceinline__ void glds(unsigned voff, const char* sbase, unsigned ldsdst) {
    asm volatile("s_mov_b32 m0, %2\n\ts_nop 0\n\tglobal_load_lds_dwordx4 %0, %1" :: "v"(voff), "s"(sbase), "s"(ldsdst) : "memory");
}
__device__ __forceinline__ void store_wt(char* p, u32x4 v) {
#define WT_MOD "sc0 sc1"
    asm volatile("global_store_dwordx4 %0, %1, off " WT_MOD "\n\ts_nop 1" :: "v"(p), "v"(v) : "memory");
}
__device__ __forceinline__ u32x4 finish_pair(f32x4 v0, f32x4 v1) {
    unsigned ax = cvt_pk_bf16(silu(v0[0]), silu(v0[1])), ay = cvt_pk_bf16(silu(v0[2]), silu(v0[3]));
    unsigned bx = cvt_pk_bf16(silu(v1[0]), silu(v1[1])), by = cvt_pk_bf16(silu(v1[2]), silu(v1[3]));
    auto rx = __builtin_amdgcn_permlane16_swap(ax, bx, false, false);
    auto ry = __builtin_amdgcn_permlane16_swap(ay, by, false, false);
    return (u32x4){rx[0], ry[0], rx[1], ry[1]};
}
#define STAMP(i) do { } while (0)
#define BAR __builtin_amdgcn_s_barrier()
#define SCHED __builtin_amdgcn_sched_barrier(0)

__device__ __forceinline__ void spin_ge(const unsigned* flag, unsigned want, unsigned* tmo, unsigned call) {
    const unsigned long long t0 = __builtin_amdgcn_s_memrealtime();
    while ((unsigned)__builtin_amdgcn_readfirstlane(__hip_atomic_load(flag, __ATOMIC_RELAXED, __HIP_MEMORY_SCOPE_SYSTEM)) < want) {
        if ((unsigned)__builtin_amdgcn_readfirstlane(__hip_atomic_load(tmo, __ATOMIC_RELAXED, __HIP_MEMORY_SCOPE_AGENT)) == call
            || __builtin_amdgcn_s_memrealtime() - t0 > DEADLINE_TICKS) {
            __hip_atomic_store(tmo, call, __ATOMIC_RELAXED, __HIP_MEMORY_SCOPE_AGENT);
            break;
        }
        __builtin_amdgcn_s_sleep(2);
    }
}

__global__ __launch_bounds__(512, 2) void gemm_a2a_kernel(Params p) {
    extern __shared__ __attribute__((aligned(16))) char smem_[];
    LAS char* smem = (LAS char*)smem_;
    const int tid = threadIdx.x, wid = __builtin_amdgcn_readfirstlane(tid >> 6), lane = tid & 63;
    const int fr = lane & 15, fq = lane >> 4;
    const int rank = p.rank;
    const int xcd = blockIdx.x & 7, c = blockIdx.x >> 3;
    const int rb0 = c & 1, slot0 = c >> 1;
    const int dest0 = (rank + 1 + (slot0 & 7)) & 7;
    const int col0 = (2 * xcd + (slot0 >> 3)) * BN;
    const int rb1 = 4 * (xcd & 3) + (c & 3), cb1 = 8 * (xcd >> 2) + (c >> 2);
    const int src1 = rb1 >> 1, h1 = rb1 & 1;
    char* const myws = p.myws;
    unsigned call;
#define PIN_CALL() do { SCHED; call = *(const __attribute__((address_space(4))) unsigned*)(size_t)(myws + WS_CALL) + 1u; asm volatile("" : "+s"(call)); } while (0)

    if (wid < 4) {
        const int wr = wid >> 1, wc = wid & 1;
        __builtin_amdgcn_s_setprio(3);
        const unsigned aoff0 = (unsigned)((wr * 64 + fr) * 128 + ((fq ^ (fr & 7)) * 16));
        const unsigned aoff1 = aoff0 ^ 64u;
        unsigned boff[2][2];
        { const int g = fq, q = fr >> 2, pp = fr & 3;
#pragma unroll
          for (int c1 = 0; c1 < 2; ++c1)
#pragma unroll
            for (int t = 0; t < 2; ++t)
              boff[c1][t] = (unsigned)(ABYTES + BN * 16 * g + 1024 * wc + 64 * (4 * t + q) + 16 * ((2 * c1 + (pp >> 1)) ^ ((2 * g + t) & 3)) + 8 * (pp & 1)); }
        f32x4 acc[NACC], oldv[NACC / 2];
#pragma unroll
        for (int i = 0; i < NACC; ++i) acc[i] = (f32x4){0.f, 0.f, 0.f, 0.f};
        s16x8 aA[MF], bA[NF], aB[MF], bB[NF];
        LAS char* xreg = smem + XOFF + wid * 8192 + lane * 16;
#define RD(a_, b_, stg, ks) do { LAS char* sb_ = smem + (stg) * STAGE; \
      _Pragma("unroll") for (int nf = 0; nf < NF; ++nf) { \
        s16x4 lo = __builtin_amdgcn_ds_read_tr16_b64_v4i16((LAS s16x4*)(sb_ + boff[nf & 1][0] + (ks) * (BN * 64) + (nf >> 1) * 512)); \
        s16x4 hi = __builtin_amdgcn_ds_read_tr16_b64_v4i16((LAS s16x4*)(sb_ + boff[nf & 1][1] + (ks) * (BN * 64) + (nf >> 1) * 512)); \
        b_[nf] = __builtin_shufflevector(lo, hi, 0, 1, 2, 3, 4, 5, 6, 7); } \
      _Pragma("unroll") for (int mf = 0; mf < MF; ++mf) a_[mf] = *(LAS s16x8*)(sb_ + ((ks) ? aoff1 : aoff0) + mf * 2048); } while (0)
#define MM(a_, b_) do { _Pragma("unroll") for (int mf = 0; mf < MF; ++mf) _Pragma("unroll") for (int nf = 0; nf < NF; ++nf) \
        acc[mf * NF + nf] = __builtin_amdgcn_mfma_f32_16x16x32_bf16(__builtin_bit_cast(bf16x8, b_[nf]), __builtin_bit_cast(bf16x8, a_[mf]), acc[mf * NF + nf], 0, 0, 0); } while (0)
#define ILV() do { _Pragma("unroll") for (int i_ = 0; i_ < MF + 2 * NF; ++i_) { __builtin_amdgcn_sched_group_barrier(0x008, 1, 0); __builtin_amdgcn_sched_group_barrier(0x100, 1, 0); } \
      __builtin_amdgcn_sched_group_barrier(0x008, NACC - (MF + 2 * NF), 0); } while (0)
#define KTILE() do { SCHED; RD(aB, bB, st, 1); MM(aA, bA); ILV(); SCHED; wait_lgkm0(); BAR; SCHED; \
      st = (st + 1) & (NS - 1); RD(aA, bA, st, 0); MM(aB, bB); ILV(); SCHED; } while (0)
        if (wid == 0) STAMP(0);
        PIN_CALL();
        BAR;
        if (wid == 0) STAMP(1);
        int st = 0;
        RD(aA, bA, 0, 0);
        for (int t = 0; t < NKT; ++t) KTILE();
        if (wid == 0) STAMP(2);
#pragma unroll
        for (int i = 0; i < NACC / 2; ++i) { *(LAS f32x4*)(xreg + i * 1024) = acc[i]; oldv[i] = acc[i + NACC / 2]; }
#pragma unroll
        for (int i = 0; i < NACC; ++i) acc[i] = (f32x4){0.f, 0.f, 0.f, 0.f};
        for (int t = 0; t <= DUMP2_T; ++t) KTILE();
#pragma unroll
        for (int i = 0; i < NACC / 2; ++i) *(LAS f32x4*)(xreg + i * 1024) = oldv[i];
        for (int t = DUMP2_T + 1; t < NKT - 4; ++t) KTILE();
        const unsigned tm = __hip_atomic_load((unsigned*)(myws + WS_TMO), __ATOMIC_RELAXED, __HIP_MEMORY_SCOPE_AGENT);
        for (int t = NKT - 4; t < NKT; ++t) KTILE();
        if (wid == 0) STAMP(3);
        const bool bad = tm == call;
        char* optr = p.myout + ((size_t)(src1 * MLOC + 256 + h1 * BM + wr * 64 + fr) * NOUT + cb1 * BN + wc * 64 + (fq & 1) * 16 + (fq >> 1) * 8) * 2;
#pragma unroll
        for (int i = 0; i < NACC / 2; ++i) {
            u32x4 wv = finish_pair(acc[2 * i], acc[2 * i + 1]);
            if (bad) wv = (u32x4){0x7fc07fc0u, 0x7fc07fc0u, 0x7fc07fc0u, 0x7fc07fc0u};
            *(u32x4*)(optr + (size_t)(i >> 1) * 16 * NOUT * 2 + (i & 1) * 64) = wv;
        }
        if (wid == 0) STAMP(4);
    } else if (wid < 6) {
        const int lw = wid - 4;
        unsigned voffA[8], voffB[4][2];
#pragma unroll
        for (int j = 0; j < 8; ++j) voffA[j] = (unsigned)((lw * 64 + j * 8 + (lane >> 3)) * (KDIM * 2) + (((lane & 7) ^ (lane >> 3)) * 16));
#pragma unroll
        for (int e = 0; e < 4; ++e)
#pragma unroll
            for (int cb = 0; cb < 2; ++cb) { const int r = (lane >> 2) & 7, j = lane >> 5, pos = lane & 3;
                voffB[e][cb] = (unsigned)((8 * (4 * lw + e) + r) * (NDIM * 2) + (4 * j + (pos ^ ((2 * e + (r >> 2)) & 3))) * 16 + cb * 128); }
        const char* A0 = p.X + (size_t)rb0 * BM * ROWB;
        const char* B0 = p.W + (size_t)(dest0 * NOUT + col0) * 2;
        const char* B1 = p.W + (size_t)(rank * NOUT + cb1 * BN) * 2;
        const unsigned lds0 = (unsigned)(size_t)smem;
        const unsigned ldA_dst = lds0 + lw * 8192, ldB_dst = lds0 + ABYTES + BN * 64 * lw;
        int ld_stage = 0;
#define LOADT(Ap, Bp, kt) do { const char* sA = (Ap) + (size_t)(kt) * 128; const char* sB = (Bp) + (size_t)(kt) * (64 * NDIM * 2); \
      const unsigned dA = ldA_dst + ld_stage * STAGE, dB = ldB_dst + ld_stage * STAGE; ld_stage = (ld_stage + 1) & (NS - 1); \
      _Pragma("unroll") for (int e = 0; e < 4; ++e) _Pragma("unroll") for (int cb = 0; cb < 2; ++cb) glds(voffB[e][cb], sB, dB + e * BN * 16 + cb * 1024); \
      _Pragma("unroll") for (int j = 0; j < 8; ++j) glds(voffA[j], sA, dA + j * 1024); } while (0)
#pragma unroll
        for (int t = 0; t < NS; ++t) LOADT(A0, B0, t);
        PIN_CALL();
        wait_vm<(NS - 1) * NLD>(); BAR;
        const char* A1 = (src1 == rank) ? p.X + (size_t)(256 + h1 * BM) * ROWB
                                        : myws + WS_G + (size_t)(call & 1) * G_PAR + (size_t)src1 * G_SRC + (size_t)h1 * BM * ROWB;
        for (int t = 0; t < NKT - NS; ++t) { wait_vm<(NS - 2) * NLD>(); BAR; LOADT(A0, B0, t + NS); }
        for (int t = 0; t < NKT; ++t) { wait_vm<(NS - 2) * NLD>(); BAR; LOADT(A1, B1, t); }
        for (int t = 0; t < NS; ++t) { wait_vm<(NS - 2) * NLD>(); BAR; LOADT(A1, B1, t); }
        wait_vm<0>();
    } else if (wid == 6) {
        u32x4 xr[8];
        const char* xsrc = p.X + (size_t)(256 + blockIdx.x) * ROWB + lane * 16;
#pragma unroll
        for (int i = 0; i < 8; ++i) xr[i] = *(const u32x4*)(xsrc + i * 1024);
        const unsigned pl = (unsigned)(rank + 1 + (lane < 7 ? lane : 0)) & 7u;
        char* peer_ws_l = p.ws[0];
#pragma unroll
        for (int q = 1; q < NRANK; ++q) peer_ws_l = (pl == (unsigned)q) ? p.ws[q] : peer_ws_l;
        unsigned* gcnt_l = (unsigned*)(peer_ws_l + WS_GCNT + (size_t)rank * NCHUNK * 128);
        asm volatile("" : "+v"(gcnt_l));
        PIN_CALL();
        const size_t goff = WS_G + (size_t)(call & 1) * G_PAR + (size_t)rank * G_SRC + (size_t)blockIdx.x * ROWB + lane * 16;
        const int qrot = (int)(blockIdx.x >> 3) % 7;
        const int soff = (int)((blockIdx.x >> 3) * 3 + (blockIdx.x & 7)) % GSTEP;
        char* optr = p.out[dest0] + ((size_t)(rank * MLOC + rb0 * BM + fr) * NOUT + col0 + (fq & 1) * 16 + (fq >> 1) * 8) * 2;
        LAS char* xd = smem + XOFF + lane * 16;
        u32x4 ob[8];
        BAR;
        int n = 0;
#define UNTIL(t_) do { const int tgt_ = (t_) < 2 * NKT ? (t_) : 2 * NKT; while (n < tgt_) { BAR; ++n; } } while (0)
#define PUBLISH_G(ch) do { wait_vm<0>(); if (lane < 7) { unsigned* a_ = gcnt_l + (ch) * 32; asm volatile("global_atomic_add %0, %1, off sc1" :: "v"(a_), "v"(1u) : "memory"); } } while (0)
#define GPIECE(i) do { for (int q = 0; q < 7; ++q) store_wt(p.ws[(rank + 1 + (q + qrot) % 7) & 7] + goff + (i) * 1024, xr[i]); } while (0)
#define CVT(j) do { ob[(j) & 7] = finish_pair(*(LAS f32x4*)(xd + ((j) >> 2) * 8192 + (2 * ((j) & 3)) * 1024), *(LAS f32x4*)(xd + ((j) >> 2) * 8192 + (2 * ((j) & 3) + 1) * 1024)); } while (0)
#define OBATCH(hf, b8) do { _Pragma("unroll") for (int k_ = 0; k_ < 8; ++k_) { const int j_ = (b8) * 8 + k_, w_ = j_ >> 2, fi_ = (hf) * 4 + (j_ & 3); \
            store_wt(optr + (size_t)((w_ >> 1) * 64 + (fi_ >> 1) * 16) * NOUT * 2 + (w_ & 1) * 128 + (fi_ & 1) * 64, ob[k_]); } } while (0)
        wait_vm<0>();
#pragma unroll
        for (int i = 0; i < 8; ++i) asm volatile("" : "+v"(xr[i]));
#pragma unroll
        for (int i = 0; i < 7; ++i) {
            UNTIL(soff + i * GSTEP);
            if (i >= 2 && (i & 1) == 0) PUBLISH_G(i / 2 - 1);
            GPIECE(i);
        }
#pragma unroll
        for (int j = 0; j < 4; ++j) { UNTIL(NKT + 1 + j + soff); CVT(j); }
        UNTIL(soff + 7 * GSTEP); GPIECE(7);
#pragma unroll
        for (int j = 4; j < 8; ++j) { UNTIL(n + 1); CVT(j); }
        UNTIL(soff + 7 * GSTEP + 8); PUBLISH_G(NCHUNK - 1);
        UNTIL(soff + 8 * GSTEP); OBATCH(0, 0);
#pragma unroll
        for (int j = 8; j < 16; ++j) { UNTIL(n + 1); CVT(j); }
        UNTIL(soff + 8 * GSTEP + OSTEP); OBATCH(0, 1);
        UNTIL(NKT + DUMP2_T + 2);
#pragma unroll
        for (int j = 0; j < 8; ++j) { CVT(j); UNTIL(n + 1); }
        UNTIL(soff + 8 * GSTEP + 2 * OSTEP); OBATCH(1, 0);
#pragma unroll
        for (int j = 8; j < 16; ++j) { CVT(j); UNTIL(n + 1); }
        UNTIL(soff + 8 * GSTEP + 3 * OSTEP); OBATCH(1, 1);
        UNTIL(2 * NKT);
        wait_vm<0>();
        STAMP(5);
        if (lane == 0) {
            if (dest0 != rank)
                __hip_atomic_fetch_add((unsigned*)(p.ws[dest0] + WS_OCNT + (size_t)rank * 128), 1u, __ATOMIC_RELAXED, __HIP_MEMORY_SCOPE_SYSTEM);
            const unsigned tk = __hip_atomic_fetch_add((unsigned*)(myws + WS_DONE), 1u, __ATOMIC_RELAXED, __HIP_MEMORY_SCOPE_AGENT);
            if ((tk + 1u) % NWG == 0u) __hip_atomic_store((unsigned*)(myws + WS_CALL), call, __ATOMIC_RELAXED, __HIP_MEMORY_SCOPE_AGENT);
        }
    } else {
        PIN_CALL();
        BAR;
        unsigned* tmo = (unsigned*)(myws + WS_TMO);
        const unsigned* gc = (const unsigned*)(myws + WS_GCNT + (size_t)src1 * NCHUNK * 128);
        int n = 0;
        for (int ch = 0; ch < NCHUNK; ++ch) {
            const int need = NKT - NS + 16 * ch;
            while (n < need - 9) { BAR; ++n; }
            if (src1 != rank) {
                unsigned v = __hip_atomic_load(gc + ch * 32, __ATOMIC_RELAXED, __HIP_MEMORY_SCOPE_SYSTEM);
                BAR; BAR; BAR; BAR; n += 4;
                if ((unsigned)__builtin_amdgcn_readfirstlane(v) < NWG * call) spin_ge(gc + ch * 32, NWG * call, tmo, call);
                BAR; BAR; BAR; BAR; n += 4;
                wait_vm<0>();
            }
        }
        while (n < 2 * NKT) { BAR; ++n; }
        if (blockIdx.x < NRANK && (int)blockIdx.x != rank) {
            spin_ge((const unsigned*)(myws + WS_OCNT + (size_t)blockIdx.x * 128), 32u * call, tmo, call);
            __builtin_amdgcn_fence(__ATOMIC_ACQUIRE, "");
            if (__hip_atomic_load(tmo, __ATOMIC_RELAXED, __HIP_MEMORY_SCOPE_AGENT) == call && lane == 0)
                *(unsigned short*)(p.myout + (size_t)(rank * MLOC) * NOUT * 2) = 0x7fc0;
        }
        STAMP(6);
    }
}

}

extern "C" void kernel_launch(int rank, int nranks, ncclComm_t comm,
                              hipStream_t stream,
                              void* const* d_in, const int* in_sizes, int n_in,
                              void* d_out, int out_size,
                              const int* in_dtypes, int out_dtype,
                              void* const* d_out_all,
                              void* const* d_ws_all, size_t ws_bytes) {
    (void)comm; (void)in_sizes; (void)n_in; (void)d_out; (void)out_size; (void)in_dtypes; (void)out_dtype;
    if (nranks != NRANK || ws_bytes < WS_NEED) { fprintf(stderr, "unsupported configuration\n"); return; }
    HIP_CHECK(hipFuncSetAttribute((const void*)gemm_a2a_kernel, hipFuncAttributeMaxDynamicSharedMemorySize, LDS_BYTES));
    Params p;
    p.X = (const char*)d_in[0];
    p.W = (const char*)d_in[1];
    for (int i = 0; i < NRANK; ++i) { p.out[i] = (char*)d_out_all[i]; p.ws[i] = (char*)d_ws_all[i]; }
    p.myws = p.ws[rank]; p.myout = p.out[rank];
    p.rank = rank;
    gemm_a2a_kernel<<<NWG, 512, LDS_BYTES, stream>>>(p);
    HIP_CHECK(hipGetLastError());
}
```

```cpp
#include <hip/hip_runtime.h>
#include <rccl/rccl.h>
#include <cstdio>

#define HIP_CHECK(expr) do { hipError_t _e = (expr); if (_e != hipSuccess) { \
    fprintf(stderr, "HIP error %s at %s:%d\n", hipGetErrorString(_e), __FILE__, __LINE__); \
    return; } } while (0)

#define LAS __attribute__((address_space(3)))
typedef short s16x4 __attribute__((ext_vector_type(4)));
typedef short s16x8 __attribute__((ext_vector_type(8)));
typedef float f32x4 __attribute__((ext_vector_type(4)));
typedef unsigned u32x4 __attribute__((ext_vector_type(4)));
typedef __bf16 bf16x8 __attribute__((ext_vector_type(8)));

namespace {

constexpr int NRANK = 8;
constexpr int MLOC = 512, KDIM = 4096, NDIM = 16384, NOUT = 2048, NKT = KDIM / 64;
constexpr int BN = 128, NF = 4, NS = 4;
constexpr int MF0 = 5, MF1 = 3, BM0 = 32 * MF0, BM1 = 32 * MF1;
constexpr int AROWS = 2 * BM0, GROWS = 2 * BM1;
constexpr int BBYTES = BN * 128, STAGE = BBYTES + BM0 * 128;
constexpr int DOFF = NS * STAGE;
constexpr int TAIL = BBYTES + BM1 * 128;
constexpr int LDS_BYTES = DOFF + 16384;
constexpr int NLD0 = 8 + BM0 / 16, NLD1 = 8 + BM1 / 16;
constexpr int NWG = 256;
constexpr int NCHUNK = 2;
constexpr size_t ROWB = (size_t)KDIM * 2;
constexpr size_t WS_CALL = 0, WS_TMO = 128, WS_DONE = 256, WS_GCNT = 1024, WS_OCNT = 8192, WS_G = 1u << 20;
constexpr size_t G_SRC = (size_t)GROWS * ROWB;
constexpr size_t G_PAR = NRANK * G_SRC;
constexpr size_t WS_NEED = WS_G + 2 * G_PAR;
constexpr unsigned long long DEADLINE_TICKS = 50000000ull;
#define GSTEP_ 11
#define OSTEP_ 12
#define OFIRST_ 68
constexpr int GSTEP = GSTEP_, OSTEP = OSTEP_, OFIRST = OFIRST_;
constexpr int DUMP2_T = 28;

struct Params { const char* X; const char* W; char* out[NRANK]; char* ws[NRANK]; char* myws; char* myout; int rank; };

template <int N> __device__ __forceinline__ void wait_vm() { asm volatile("s_waitcnt vmcnt(%0)" :: "n"(N) : "memory"); }
__device__ __forceinline__ void wait_lgkm0() { asm volatile("s_waitcnt lgkmcnt(0)" ::: "memory"); }
__device__ __forceinline__ unsigned cvt_pk_bf16(float lo, float hi) { unsigned r; asm("v_cvt_pk_bf16_f32 %0, %1, %2" : "=v"(r) : "v"(lo), "v"(hi)); return r; }
__device__ __forceinline__ float silu(float y) {
    float c = __builtin_amdgcn_fmed3f(y, -60.f, 60.f);
    float e = __builtin_amdgcn_exp2f(c * -1.4426950408889634f);
    return y * __builtin_amdgcn_rcpf(1.0f + e);
}
__device__ __forceinline__ void glds(unsigned voff, const char* sbase, unsigned ldsdst) {
    asm volatile("s_mov_b32 m0, %2\n\ts_nop 0\n\tglobal_load_lds_dwordx4 %0, %1" :: "v"(voff), "s"(sbase), "s"(ldsdst) : "memory");
}
__device__ __forceinline__ void store_wt(char* p, u32x4 v) {
    asm volatile("global_store_dwordx4 %0, %1, off sc0 sc1\n\ts_nop 1" :: "v"(p), "v"(v) : "memory");
}
__device__ __forceinline__ u32x4 finish_pair(f32x4 v0, f32x4 v1) {
    unsigned ax = cvt_pk_bf16(silu(v0[0]), silu(v0[1])), ay = cvt_pk_bf16(silu(v0[2]), silu(v0[3]));
    unsigned bx = cvt_pk_bf16(silu(v1[0]), silu(v1[1])), by = cvt_pk_bf16(silu(v1[2]), silu(v1[3]));
    auto rx = __builtin_amdgcn_permlane16_swap(ax, bx, false, false);
    auto ry = __builtin_amdgcn_permlane16_swap(ay, by, false, false);
    return (u32x4){rx[0], ry[0], rx[1], ry[1]};
}
__device__ __forceinline__ constexpr int dump_slot(int w, int s) { return s == 0 ? DOFF + w * 4096 : w * STAGE + TAIL + (s - 1) * 4096; }
#define STAMP(i) do { } while (0)
#define BAR __builtin_amdgcn_s_barrier()
#define SCHED __builtin_amdgcn_sched_barrier(0)

__device__ __forceinline__ void spin_ge(const unsigned* flag, unsigned want, unsigned* tmo, unsigned call) {
    const unsigned long long t0 = __builtin_amdgcn_s_memrealtime();
    while ((unsigned)__builtin_amdgcn_readfirstlane(__hip_atomic_load(flag, __ATOMIC_RELAXED, __HIP_MEMORY_SCOPE_SYSTEM)) < want) {
        if ((unsigned)__builtin_amdgcn_readfirstlane(__hip_atomic_load(tmo, __ATOMIC_RELAXED, __HIP_MEMORY_SCOPE_AGENT)) == call
            || __builtin_amdgcn_s_memrealtime() - t0 > DEADLINE_TICKS) {
            __hip_atomic_store(tmo, call, __ATOMIC_RELAXED, __HIP_MEMORY_SCOPE_AGENT);
            break;
        }
        __builtin_amdgcn_s_sleep(2);
    }
}

__global__ __launch_bounds__(512, 2) void gemm_a2a_kernel(Params p) {
    extern __shared__ __attribute__((aligned(16))) char smem_[];
    LAS char* smem = (LAS char*)smem_;
    const int tid = threadIdx.x, wid = __builtin_amdgcn_readfirstlane(tid >> 6), lane = tid & 63;
    const int fr = lane & 15, fq = lane >> 4;
    const int rank = p.rank;
    const int xcd = blockIdx.x & 7, c = blockIdx.x >> 3;
    const int rb0 = c & 1, slot0 = c >> 1;
    const int dest0 = (rank + 1 + (slot0 & 7)) & 7;
    const int col0 = (2 * xcd + (slot0 >> 3)) * BN;
    const int rb1 = 4 * (xcd & 3) + (c & 3), cb1 = 8 * (xcd >> 2) + (c >> 2);
    const int src1 = rb1 >> 1, h1 = rb1 & 1;
    char* const myws = p.myws;
    unsigned call;
#define PIN_CALL() do { SCHED; call = *(const __attribute__((address_space(4))) unsigned*)(size_t)(myws + WS_CALL) + 1u; asm volatile("" : "+s"(call)); } while (0)

    if (wid < 4) {
        const int wr = wid >> 1, wc = wid & 1;
        __builtin_amdgcn_s_setprio(3);
        const unsigned asw = (unsigned)(BBYTES + fr * 128 + ((fq ^ (fr & 7)) * 16));
        unsigned aoff0 = asw + wr * (BM0 / 2) * 128, aoff1 = aoff0 ^ 64u;
        unsigned boff[2][2];
        { const int g = fq, q = fr >> 2, pp = fr & 3;
#pragma unroll
          for (int c1 = 0; c1 < 2; ++c1)
#pragma unroll
            for (int t = 0; t < 2; ++t)
              boff[c1][t] = (unsigned)(BN * 16 * g + 1024 * wc + 64 * (4 * t + q) + 16 * ((2 * c1 + (pp >> 1)) ^ ((2 * g + t) & 3)) + 8 * (pp & 1)); }
        f32x4 acc[MF0 * NF], oldv[2 * NF];
#pragma unroll
        for (int i = 0; i < MF0 * NF; ++i) acc[i] = (f32x4){0.f, 0.f, 0.f, 0.f};
        s16x8 aA[MF0], bA[NF], aB[MF0], bB[NF];
#define RD(MF_, a_, b_, stg, ks) do { LAS char* sb_ = smem + (stg) * STAGE; \
      _Pragma("unroll") for (int nf = 0; nf < NF; ++nf) { \
        s16x4 lo = __builtin_amdgcn_ds_read_tr16_b64_v4i16((LAS s16x4*)(sb_ + boff[nf & 1][0] + (ks) * (BN * 64) + (nf >> 1) * 512)); \
        s16x4 hi = __builtin_amdgcn_ds_read_tr16_b64_v4i16((LAS s16x4*)(sb_ + boff[nf & 1][1] + (ks) * (BN * 64) + (nf >> 1) * 512)); \
        b_[nf] = __builtin_shufflevector(lo, hi, 0, 1, 2, 3, 4, 5, 6, 7); } \
      _Pragma("unroll") for (int mf = 0; mf < (MF_); ++mf) a_[mf] = *(LAS s16x8*)(sb_ + ((ks) ? aoff1 : aoff0) + mf * 2048); } while (0)
#define MM(MF_, a_, b_) do { _Pragma("unroll") for (int mf = 0; mf < (MF_); ++mf) _Pragma("unroll") for (int nf = 0; nf < NF; ++nf) \
        acc[mf * NF + nf] = __builtin_amdgcn_mfma_f32_16x16x32_bf16(__builtin_bit_cast(bf16x8, b_[nf]), __builtin_bit_cast(bf16x8, a_[mf]), acc[mf * NF + nf], 0, 0, 0); } while (0)
#define ILV(NM_, NR_) do { _Pragma("unroll") for (int i_ = 0; i_ < (NR_); ++i_) { __builtin_amdgcn_sched_group_barrier(0x008, 1, 0); __builtin_amdgcn_sched_group_barrier(0x100, 1, 0); } \
      __builtin_amdgcn_sched_group_barrier(0x008, (NM_) - (NR_), 0); } while (0)
#define KTILE2(MF_, MFN_, PRE) do { SCHED; RD(MF_, aB, bB, st, 1); MM(MF_, aA, bA); ILV((MF_) * NF, (MF_) + 2 * NF); SCHED; wait_lgkm0(); BAR; SCHED; \
      st = (st + 1) & (NS - 1); PRE; RD(MFN_, aA, bA, st, 0); MM(MF_, aB, bB); ILV((MF_) * NF, (MFN_) + 2 * NF); SCHED; } while (0)
#define KTILE(MF_) KTILE2(MF_, MF_, )
        if (wid == 0) STAMP(0);
        PIN_CALL();
        BAR;
        if (wid == 0) STAMP(1);
        int st = 0;
        RD(MF0, aA, bA, 0, 0);
        for (int t = 0; t < NKT - 1; ++t) KTILE(MF0);
        KTILE2(MF0, MF1, aoff0 = asw + wr * (BM1 / 2) * 128; aoff1 = aoff0 ^ 64u);
        if (wid == 0) STAMP(2);
        {
            LAS char* d0 = smem + DOFF + wid * 4096 + lane * 16;
            LAS char* d1 = smem + wid * STAGE + TAIL + lane * 16;
#pragma unroll
            for (int i = 0; i < NF; ++i) *(LAS f32x4*)(d0 + i * 1024) = acc[i];
#pragma unroll
            for (int i = 0; i < 2 * NF; ++i) { *(LAS f32x4*)(d1 + i * 1024) = acc[NF + i]; oldv[i] = acc[3 * NF + i]; }
#pragma unroll
            for (int i = 0; i < MF1 * NF; ++i) acc[i] = (f32x4){0.f, 0.f, 0.f, 0.f};
            for (int t = 0; t <= DUMP2_T; ++t) KTILE(MF1);
#pragma unroll
            for (int i = 0; i < 2 * NF; ++i) *(LAS f32x4*)(d1 + i * 1024) = oldv[i];
        }
        for (int t = DUMP2_T + 1; t < NKT - 4; ++t) KTILE(MF1);
        const unsigned tm = __hip_atomic_load((unsigned*)(myws + WS_TMO), __ATOMIC_RELAXED, __HIP_MEMORY_SCOPE_AGENT);
        for (int t = NKT - 4; t < NKT; ++t) KTILE(MF1);
        if (wid == 0) STAMP(3);
        const bool bad = tm == call;
        char* optr = p.myout + ((size_t)(src1 * MLOC + AROWS + h1 * BM1 + wr * (BM1 / 2) + fr) * NOUT + cb1 * BN + wc * 64 + (fq & 1) * 16 + (fq >> 1) * 8) * 2;
#pragma unroll
        for (int i = 0; i < MF1 * NF / 2; ++i) {
            u32x4 wv = finish_pair(acc[2 * i], acc[2 * i + 1]);
            if (bad) wv = (u32x4){0x7fc07fc0u, 0x7fc07fc0u, 0x7fc07fc0u, 0x7fc07fc0u};
            *(u32x4*)(optr + (size_t)(i >> 1) * 16 * NOUT * 2 + (i & 1) * 64) = wv;
        }
        if (wid == 0) STAMP(4);
    } else if (wid < 6) {
        const int lw = wid - 4;
        unsigned voffA[BM0 / 16], voffB[4][2];
#pragma unroll
        for (int j = 0; j < BM0 / 16; ++j) voffA[j] = (unsigned)((16 * j + 8 * lw + (lane >> 3)) * (KDIM * 2) + (((lane & 7) ^ (lane >> 3)) * 16));
#pragma unroll
        for (int e = 0; e < 4; ++e)
#pragma unroll
            for (int cb = 0; cb < 2; ++cb) { const int r = (lane >> 2) & 7, j = lane >> 5, pos = lane & 3;
                voffB[e][cb] = (unsigned)((8 * (4 * lw + e) + r) * (NDIM * 2) + (4 * j + (pos ^ ((2 * e + (r >> 2)) & 3))) * 16 + cb * 128); }
        const char* A0 = p.X + (size_t)rb0 * BM0 * ROWB;
        const char* B0 = p.W + (size_t)(dest0 * NOUT + col0) * 2;
        const char* B1 = p.W + (size_t)(rank * NOUT + cb1 * BN) * 2;
        const unsigned lds0 = (unsigned)(size_t)smem;
        const unsigned ldA_dst = lds0 + BBYTES + lw * 1024, ldB_dst = lds0 + BN * 64 * lw;
        int ld_stage = 0;
#define LOADT(BM_, Ap, Bp, kt) do { const char* sA = (Ap) + (size_t)(kt) * 128; const char* sB = (Bp) + (size_t)(kt) * (64 * NDIM * 2); \
      const unsigned dA = ldA_dst + ld_stage * STAGE, dB = ldB_dst + ld_stage * STAGE; ld_stage = (ld_stage + 1) & (NS - 1); \
      _Pragma("unroll") for (int e = 0; e < 4; ++e) _Pragma("unroll") for (int cb = 0; cb < 2; ++cb) glds(voffB[e][cb], sB, dB + e * BN * 16 + cb * 1024); \
      _Pragma("unroll") for (int j = 0; j < (BM_) / 16; ++j) glds(voffA[j], sA, dA + j * 2048); } while (0)
#pragma unroll
        for (int t = 0; t < NS; ++t) LOADT(BM0, A0, B0, t);
        PIN_CALL();
        wait_vm<3 * NLD0>(); BAR;
        const char* A1 = (src1 == rank) ? p.X + (size_t)(AROWS + h1 * BM1) * ROWB
                                        : myws + WS_G + (size_t)(call & 1) * G_PAR + (size_t)src1 * G_SRC + (size_t)h1 * BM1 * ROWB;
        for (int t = 0; t < NKT - NS; ++t) { wait_vm<2 * NLD0>(); BAR; LOADT(BM0, A0, B0, t + NS); }
        wait_vm<2 * NLD0>(); BAR; LOADT(BM1, A1, B1, 0);
        wait_vm<NLD0 + NLD1>(); BAR; LOADT(BM1, A1, B1, 1);
        for (int t = 2; t < NKT; ++t) { wait_vm<2 * NLD1>(); BAR; LOADT(BM1, A1, B1, t); }
        for (int t = 0; t < NS; ++t) { wait_vm<2 * NLD1>(); BAR; LOADT(BM1, A1, B1, t); }
        wait_vm<0>();
    } else if (wid == 6) {
        u32x4 xr[6];
        unsigned poff[3];
#pragma unroll
        for (int i = 0; i < 3; ++i) { const unsigned q = blockIdx.x * 3 + i; poff[i] = (q >> 2) * (unsigned)ROWB + (q & 3) * 1024 + lane * 16; }
#pragma unroll
        for (int i = 0; i < 6; ++i) xr[i] = *(const u32x4*)(p.X + (size_t)AROWS * ROWB + poff[i % 3] + (i / 3) * 4096);
        const unsigned pl = (unsigned)(rank + 1 + (lane < 7 ? lane : 0)) & 7u;
        char* peer_ws_l = p.ws[0];
#pragma unroll
        for (int q = 1; q < NRANK; ++q) peer_ws_l = (pl == (unsigned)q) ? p.ws[q] : peer_ws_l;
        unsigned* gcnt_l = (unsigned*)(peer_ws_l + WS_GCNT + (size_t)rank * NCHUNK * 128);
        asm volatile("" : "+v"(gcnt_l));
        PIN_CALL();
        const size_t goff = WS_G + (size_t)(call & 1) * G_PAR + (size_t)rank * G_SRC;
        const int qrot = (int)(blockIdx.x >> 3) % 7;
        const int sidx = (int)((blockIdx.x >> 3) * 3 + (blockIdx.x & 7));
        const int soff = sidx % GSTEP, soff2 = sidx % OSTEP;
        char* optr = p.out[dest0] + ((size_t)(rank * MLOC + rb0 * BM0 + fr) * NOUT + col0 + (fq & 1) * 16 + (fq >> 1) * 8) * 2;
        LAS char* xd = smem + lane * 16;
        u32x4 ob[40];
        BAR;
        int n = 0;
#define UNTIL(t_) do { const int tgt_ = (t_) < 2 * NKT ? (t_) : 2 * NKT; while (n < tgt_) { BAR; ++n; } } while (0)
#define PUBLISH_G(ch) do { wait_vm<0>(); if (lane < 7) { unsigned* a_ = gcnt_l + (ch) * 32; asm volatile("global_atomic_add %0, %1, off sc1" :: "v"(a_), "v"(1u) : "memory"); } } while (0)
#define GPIECE(i) do { for (int q = 0; q < 7; ++q) store_wt(p.ws[(rank + 1 + (q + qrot) % 7) & 7] + goff + poff[(i) % 3] + ((i) / 3) * 4096, xr[i]); } while (0)
#define CVT(j) do { constexpr int mf_ = (j) / 8, w_ = ((j) >> 1) & 3, o_ = dump_slot(w_, mf_ < 3 ? mf_ : mf_ - 2) + ((j) & 1) * 2048; \
            ob[j] = finish_pair(*(LAS f32x4*)(xd + o_), *(LAS f32x4*)(xd + o_ + 1024)); } while (0)
#define OB 4
#define OST(j_) store_wt(optr + (size_t)(((((j_) >> 1) & 3) >> 1) * (BM0 / 2) + ((j_) / 8) * 16) * NOUT * 2 + (((j_) >> 1) & 1) * 128 + ((j_) & 1) * 64, ob[j_])
        int nb = 0;
        bool gpub = false;
        const int so2 = sidx % ((OSTEP * OB + 7) / 8);
#define TRY_BURST(avail) do { \
            if (!gpub && n >= soff + 5 * GSTEP + 8) { PUBLISH_G(1); gpub = true; } \
            if ((nb + 1) * OB <= (avail) && n >= OFIRST + (nb * OSTEP * OB) / 8 + so2) { \
                _Pragma("unroll") for (int b_ = 0; b_ < (avail) / OB; ++b_) if (nb == b_) { _Pragma("unroll") for (int k_ = 0; k_ < OB; ++k_) OST(b_ * OB + k_); } \
                ++nb; } } while (0)
        wait_vm<0>();
#pragma unroll
        for (int i = 0; i < 6; ++i) asm volatile("" : "+v"(xr[i]));
#pragma unroll
        for (int i = 0; i < 6; ++i) {
            UNTIL(soff + i * GSTEP);
            if (i == 3) PUBLISH_G(0);
            GPIECE(i);
        }
        UNTIL(NKT + 1);
#define CV1(j) do { CVT(j); TRY_BURST((j) + 1); UNTIL(n + 1); } while (0)
#define CV8(b) do { CV1((b) * 8 + 0); CV1((b) * 8 + 1); CV1((b) * 8 + 2); CV1((b) * 8 + 3); CV1((b) * 8 + 4); CV1((b) * 8 + 5); CV1((b) * 8 + 6); CV1((b) * 8 + 7); } while (0)
        CV8(0); CV8(1); CV8(2);
        while (n < NKT + DUMP2_T + 2) { TRY_BURST(24); BAR; ++n; }
        CV8(3); CV8(4);
        while (n < 2 * NKT) { TRY_BURST(40); BAR; ++n; }
        n = 1 << 20;
        while (nb * OB < 40 || !gpub) TRY_BURST(40);
        wait_vm<0>();
        STAMP(5);
        if (lane == 0) {
            if (dest0 != rank)
                __hip_atomic_fetch_add((unsigned*)(p.ws[dest0] + WS_OCNT + (size_t)rank * 128), 1u, __ATOMIC_RELAXED, __HIP_MEMORY_SCOPE_SYSTEM);
            const unsigned tk = __hip_atomic_fetch_add((unsigned*)(myws + WS_DONE), 1u, __ATOMIC_RELAXED, __HIP_MEMORY_SCOPE_AGENT);
            if ((tk + 1u) % NWG == 0u) __hip_atomic_store((unsigned*)(myws + WS_CALL), call, __ATOMIC_RELAXED, __HIP_MEMORY_SCOPE_AGENT);
        }
    } else {
        PIN_CALL();
        BAR;
        unsigned* tmo = (unsigned*)(myws + WS_TMO);
        const unsigned* gc = (const unsigned*)(myws + WS_GCNT + (size_t)src1 * NCHUNK * 128);
        int n = 0;
        for (int ch = 0; ch < NCHUNK; ++ch) {
            const int need = NKT - NS + (NKT / NCHUNK) * ch;
            while (n < need - 6) { BAR; ++n; }
            if (src1 != rank) {
                unsigned v = __hip_atomic_load(gc + ch * 32, __ATOMIC_RELAXED, __HIP_MEMORY_SCOPE_SYSTEM);
                BAR; BAR; BAR; BAR; BAR; n += 5;
                if ((unsigned)__builtin_amdgcn_readfirstlane(v) < NWG * call) spin_ge(gc + ch * 32, NWG * call, tmo, call);
                asm volatile("" ::: "memory");
            }
        }
        while (n < 2 * NKT) { BAR; ++n; }
        if (blockIdx.x < NRANK && (int)blockIdx.x != rank) {
            spin_ge((const unsigned*)(myws + WS_OCNT + (size_t)blockIdx.x * 128), 32u * call, tmo, call);
            __builtin_amdgcn_fence(__ATOMIC_ACQUIRE, "");
            if (__hip_atomic_load(tmo, __ATOMIC_RELAXED, __HIP_MEMORY_SCOPE_AGENT) == call && lane == 0)
                *(unsigned short*)(p.myout + (size_t)(rank * MLOC) * NOUT * 2) = 0x7fc0;
        }
        STAMP(6);
    }
}

}

extern "C" void kernel_launch(int rank, int nranks, ncclComm_t comm,
                              hipStream_t stream,
                              void* const* d_in, const int* in_sizes, int n_in,
                              void* d_out, int out_size,
                              const int* in_dtypes, int out_dtype,
                              void* const* d_out_all,
                              void* const* d_ws_all, size_t ws_bytes) {
    (void)comm; (void)in_sizes; (void)n_in; (void)d_out; (void)out_size; (void)in_dtypes; (void)out_dtype;
    if (nranks != NRANK || ws_bytes < WS_NEED) { fprintf(stderr, "unsupported configuration\n"); return; }
    HIP_CHECK(hipFuncSetAttribute((const void*)gemm_a2a_kernel, hipFuncAttributeMaxDynamicSharedMemorySize, LDS_BYTES));
    Params p;
    p.X = (const char*)d_in[0];
    p.W = (const char*)d_in[1];
    for (int i = 0; i < NRANK; ++i) { p.out[i] = (char*)d_out_all[i]; p.ws[i] = (char*)d_ws_all[i]; }
    p.myws = p.ws[rank]; p.myout = p.out[rank];
    p.rank = rank;
    gemm_a2a_kernel<<<NWG, 512, LDS_BYTES, stream>>>(p);
    HIP_CHECK(hipGetLastError());
}
```

```cpp
#include <hip/hip_runtime.h>
#include <rccl/rccl.h>
#include <cstdio>

#define HIP_CHECK(expr) do { hipError_t _e = (expr); if (_e != hipSuccess) { \
    fprintf(stderr, "HIP error %s at %s:%d\n", hipGetErrorString(_e), __FILE__, __LINE__); \
    return; } } while (0)

#define LAS __attribute__((address_space(3)))
typedef short s16x4 __attribute__((ext_vector_type(4)));
typedef short s16x8 __attribute__((ext_vector_type(8)));
typedef float f32x4 __attribute__((ext_vector_type(4)));
typedef unsigned u32x4 __attribute__((ext_vector_type(4)));
typedef __bf16 bf16x8 __attribute__((ext_vector_type(8)));

namespace {

constexpr int NRANK = 8;
constexpr int MLOC = 512, KDIM = 4096, NDIM = 16384, NOUT = 2048, NKT = KDIM / 64;
constexpr int BN = 128, NF = 4, NS = 4;
constexpr int MF0 = 5, MF1 = 4, BM0 = 32 * MF0, BM1 = 32 * MF1;
constexpr int BROWS = 96, XROWS = 20;
constexpr int AROWS = 2 * BM0, GROWS = 2 * BROWS;
constexpr int BBYTES = BN * 128, STAGE = BBYTES + BM0 * 128;
constexpr int DOFF = NS * STAGE;
constexpr int TAIL = BBYTES + BM1 * 128;
constexpr int LDS_BYTES = DOFF + 16384;
constexpr int NLD0 = 8 + BM0 / 16, NLD1 = 8 + BM1 / 16;
constexpr int NWG = 256;
constexpr int NCHUNK = 2;
constexpr size_t ROWB = (size_t)KDIM * 2;
constexpr size_t WS_CALL = 0, WS_TMO = 128, WS_DONE = 256, WS_GCNT = 1024, WS_OCNT = 8192, WS_G = 1u << 20;
constexpr size_t G_SRC = (size_t)GROWS * ROWB;
constexpr size_t G_PAR = NRANK * G_SRC;
constexpr size_t WS_NEED = WS_G + 2 * G_PAR;
constexpr unsigned long long DEADLINE_TICKS = 50000000ull;
#define GSTEP_ 11
#define OSTEP_ 13
#define OFIRST_ 68
constexpr int GSTEP = GSTEP_, OSTEP = OSTEP_, OFIRST = OFIRST_;
constexpr int DUMP2_T = 18, DUMP3_T = 37;

struct Params { const char* X; const char* W; char* out[NRANK]; char* ws[NRANK]; char* myws; char* myout; int rank; };

template <int N> __device__ __forceinline__ void wait_vm() { asm volatile("s_waitcnt vmcnt(%0)" :: "n"(N) : "memory"); }
__device__ __forceinline__ void wait_lgkm0() { asm volatile("s_waitcnt lgkmcnt(0)" ::: "memory"); }
__device__ __forceinline__ unsigned cvt_pk_bf16(float lo, float hi) { unsigned r; asm("v_cvt_pk_bf16_f32 %0, %1, %2" : "=v"(r) : "v"(lo), "v"(hi)); return r; }
__device__ __forceinline__ float silu(float y) {
    float c = __builtin_amdgcn_fmed3f(y, -60.f, 60.f);
    float e = __builtin_amdgcn_exp2f(c * -1.4426950408889634f);
    return y * __builtin_amdgcn_rcpf(1.0f + e);
}
__device__ __forceinline__ void glds(unsigned voff, const char* sbase, unsigned ldsdst) {
    asm volatile("s_mov_b32 m0, %2\n\ts_nop 0\n\tglobal_load_lds_dwordx4 %0, %1" :: "v"(voff), "s"(sbase), "s"(ldsdst) : "memory");
}
__device__ __forceinline__ void store_wt(char* p, u32x4 v) {
    asm volatile("global_store_dwordx4 %0, %1, off sc0 sc1\n\ts_nop 1" :: "v"(p), "v"(v) : "memory");
}
__device__ __forceinline__ u32x4 finish_pair(f32x4 v0, f32x4 v1) {
    unsigned ax = cvt_pk_bf16(silu(v0[0]), silu(v0[1])), ay = cvt_pk_bf16(silu(v0[2]), silu(v0[3]));
    unsigned bx = cvt_pk_bf16(silu(v1[0]), silu(v1[1])), by = cvt_pk_bf16(silu(v1[2]), silu(v1[3]));
    auto rx = __builtin_amdgcn_permlane16_swap(ax, bx, false, false);
    auto ry = __builtin_amdgcn_permlane16_swap(ay, by, false, false);
    return (u32x4){rx[0], ry[0], rx[1], ry[1]};
}
__device__ __forceinline__ constexpr int dump_slot(int w, int s) { return s == 0 ? DOFF + w * 4096 : w * STAGE + TAIL; }
#define STAMP(i) do { } while (0)
#define BAR __builtin_amdgcn_s_barrier()
#define SCHED __builtin_amdgcn_sched_barrier(0)

__device__ __forceinline__ void spin_ge(const unsigned* flag, unsigned want, unsigned* tmo, unsigned call) {
    const unsigned long long t0 = __builtin_amdgcn_s_memrealtime();
    while ((unsigned)__builtin_amdgcn_readfirstlane(__hip_atomic_load(flag, __ATOMIC_RELAXED, __HIP_MEMORY_SCOPE_SYSTEM)) < want) {
        if ((unsigned)__builtin_amdgcn_readfirstlane(__hip_atomic_load(tmo, __ATOMIC_RELAXED, __HIP_MEMORY_SCOPE_AGENT)) == call
            || __builtin_amdgcn_s_memrealtime() - t0 > DEADLINE_TICKS) {
            __hip_atomic_store(tmo, call, __ATOMIC_RELAXED, __HIP_MEMORY_SCOPE_AGENT);
            break;
        }
        __builtin_amdgcn_s_sleep(2);
    }
}

__global__ __launch_bounds__(512, 2) void gemm_a2a_kernel(Params p) {
    extern __shared__ __attribute__((aligned(16))) char smem_[];
    LAS char* smem = (LAS char*)smem_;
    const int tid = threadIdx.x, wid = __builtin_amdgcn_readfirstlane(tid >> 6), lane = tid & 63;
    const int fr = lane & 15, fq = lane >> 4;
    const int rank = p.rank;
    const int xcd = blockIdx.x & 7, c = blockIdx.x >> 3;
    const int rb0 = c & 1, slot0 = c >> 1;
    const int dest0 = (rank + 1 + (slot0 & 7)) & 7;
    const int col0 = (2 * xcd + (slot0 >> 3)) * BN;
    const int rb1 = 4 * (xcd & 3) + (c & 3), cb1 = 8 * (xcd >> 2) + (c >> 2);
    const int src1 = rb1 >> 1, h1 = rb1 & 1;
    char* const myws = p.myws;
    unsigned call;
    const bool isC = dest0 == rank;
#define PIN_CALL() do { SCHED; call = *(const __attribute__((address_space(4))) unsigned*)(size_t)(myws + WS_CALL) + 1u; asm volatile("" : "+s"(call)); } while (0)

    if (isC) {
        PIN_CALL();
        const unsigned ci = (unsigned)(xcd * 4 + rb0 + 2 * (slot0 >> 3)) * 8u + (unsigned)wid;
        u32x4 xr[6]; unsigned poff[3];
#pragma unroll
        for (int i = 0; i < 3; ++i) { const unsigned q = ci * 3 + i; poff[i] = (q >> 2) * (unsigned)ROWB + (q & 3) * 1024 + lane * 16; }
#pragma unroll
        for (int i = 0; i < 6; ++i) xr[i] = *(const u32x4*)(p.X + (size_t)AROWS * ROWB + poff[i % 3] + (i / 3) * 4096);
        const unsigned pl = (unsigned)(rank + 1 + (lane < 7 ? lane : 0)) & 7u;
        char* peer_ws_l = p.ws[0];
#pragma unroll
        for (int q = 1; q < NRANK; ++q) peer_ws_l = (pl == (unsigned)q) ? p.ws[q] : peer_ws_l;
        unsigned* gcnt_l = (unsigned*)(peer_ws_l + WS_GCNT + (size_t)rank * NCHUNK * 128);
        asm volatile("" : "+v"(gcnt_l));
        const size_t goff = WS_G + (size_t)(call & 1) * G_PAR + (size_t)rank * G_SRC;
        const int qrot = (int)(ci % 7u);
        wait_vm<0>();
#pragma unroll
        for (int i = 0; i < 6; ++i) asm volatile("" : "+v"(xr[i]));
#pragma unroll
        for (int i = 0; i < 6; ++i) {
            for (int q = 0; q < 7; ++q) store_wt(p.ws[(rank + 1 + (q + qrot) % 7) & 7] + goff + poff[i % 3] + (i / 3) * 4096, xr[i]);
            if (i == 3) { wait_vm<7>(); if (lane < 7) asm volatile("global_atomic_add %0, %1, off sc1" :: "v"(gcnt_l), "v"(1u) : "memory"); }
        }
        wait_vm<0>();
        if (lane < 7) { unsigned* a_ = gcnt_l + 32; asm volatile("global_atomic_add %0, %1, off sc1" :: "v"(a_), "v"(1u) : "memory"); }
    }

    if (wid < 4) {
        const int wr = wid >> 1, wc = wid & 1;
        __builtin_amdgcn_s_setprio(3);
        const unsigned asw = (unsigned)(BBYTES + fr * 128 + ((fq ^ (fr & 7)) * 16));
        unsigned aoff0 = asw + wr * (BM0 / 2) * 128, aoff1 = aoff0 ^ 64u;
        unsigned boff[2][2];
        { const int g = fq, q = fr >> 2, pp = fr & 3;
#pragma unroll
          for (int c1 = 0; c1 < 2; ++c1)
#pragma unroll
            for (int t = 0; t < 2; ++t)
              boff[c1][t] = (unsigned)(BN * 16 * g + 1024 * wc + 64 * (4 * t + q) + 16 * ((2 * c1 + (pp >> 1)) ^ ((2 * g + t) & 3)) + 8 * (pp & 1)); }
        f32x4 acc[MF0 * NF], oldv[3 * NF];
#pragma unroll
        for (int i = 0; i < MF0 * NF; ++i) acc[i] = (f32x4){0.f, 0.f, 0.f, 0.f};
        s16x8 aA[MF0], bA[NF], aB[MF0], bB[NF];
#define RD(MF_, a_, b_, stg, ks) do { LAS char* sb_ = smem + (stg) * STAGE; \
      _Pragma("unroll") for (int nf = 0; nf < NF; ++nf) { \
        s16x4 lo = __builtin_amdgcn_ds_read_tr16_b64_v4i16((LAS s16x4*)(sb_ + boff[nf & 1][0] + (ks) * (BN * 64) + (nf >> 1) * 512)); \
        s16x4 hi = __builtin_amdgcn_ds_read_tr16_b64_v4i16((LAS s16x4*)(sb_ + boff[nf & 1][1] + (ks) * (BN * 64) + (nf >> 1) * 512)); \
        b_[nf] = __builtin_shufflevector(lo, hi, 0, 1, 2, 3, 4, 5, 6, 7); } \
      _Pragma("unroll") for (int mf = 0; mf < (MF_); ++mf) a_[mf] = *(LAS s16x8*)(sb_ + ((ks) ? aoff1 : aoff0) + mf * 2048); } while (0)
#define MM(MF_, a_, b_) do { _Pragma("unroll") for (int mf = 0; mf < (MF_); ++mf) _Pragma("unroll") for (int nf = 0; nf < NF; ++nf) \
        acc[mf * NF + nf] = __builtin_amdgcn_mfma_f32_16x16x32_bf16(__builtin_bit_cast(bf16x8, b_[nf]), __builtin_bit_cast(bf16x8, a_[mf]), acc[mf * NF + nf], 0, 0, 0); } while (0)
#define ILV(NM_, NR_) do { _Pragma("unroll") for (int i_ = 0; i_ < (NR_); ++i_) { __builtin_amdgcn_sched_group_barrier(0x008, 1, 0); __builtin_amdgcn_sched_group_barrier(0x100, 1, 0); } \
      __builtin_amdgcn_sched_group_barrier(0x008, (NM_) - (NR_), 0); } while (0)
#define KTILE2(MF_, MFN_, PRE) do { SCHED; RD(MF_, aB, bB, st, 1); MM(MF_, aA, bA); ILV((MF_) * NF, (MF_) + 2 * NF); SCHED; wait_lgkm0(); BAR; SCHED; \
      st = (st + 1) & (NS - 1); PRE; RD(MFN_, aA, bA, st, 0); MM(MF_, aB, bB); ILV((MF_) * NF, (MFN_) + 2 * NF); SCHED; } while (0)
#define KTILE(MF_) KTILE2(MF_, MF_, )
        if (wid == 0) STAMP(0);
        PIN_CALL();
        int st = 0;
        if (isC) {
            BAR;
            BAR;
            aoff0 = asw + wr * (BM1 / 2) * 128; aoff1 = aoff0 ^ 64u;
            RD(MF1, aA, bA, 0, 0);
            for (int t = 0; t < NKT - 4; ++t) KTILE(MF1);
        } else {
            BAR;
            if (wid == 0) STAMP(1);
            RD(MF0, aA, bA, 0, 0);
            for (int t = 0; t < NKT - 1; ++t) KTILE(MF0);
            KTILE2(MF0, MF1, aoff0 = asw + wr * (BM1 / 2) * 128; aoff1 = aoff0 ^ 64u);
            if (wid == 0) STAMP(2);
            LAS char* d0 = smem + DOFF + wid * 4096 + lane * 16;
            LAS char* d1 = smem + wid * STAGE + TAIL + lane * 16;
#pragma unroll
            for (int i = 0; i < NF; ++i) { *(LAS f32x4*)(d0 + i * 1024) = acc[i]; *(LAS f32x4*)(d1 + i * 1024) = acc[NF + i]; }
#pragma unroll
            for (int i = 0; i < 3 * NF; ++i) oldv[i] = acc[2 * NF + i];
#pragma unroll
            for (int i = 0; i < MF1 * NF; ++i) acc[i] = (f32x4){0.f, 0.f, 0.f, 0.f};
            for (int t = 0; t <= DUMP2_T; ++t) KTILE(MF1);
#pragma unroll
            for (int i = 0; i < NF; ++i) { *(LAS f32x4*)(d0 + i * 1024) = oldv[i]; *(LAS f32x4*)(d1 + i * 1024) = oldv[NF + i]; }
            for (int t = DUMP2_T + 1; t <= DUMP3_T; ++t) KTILE(MF1);
#pragma unroll
            for (int i = 0; i < NF; ++i) *(LAS f32x4*)(d0 + i * 1024) = oldv[2 * NF + i];
            for (int t = DUMP3_T + 1; t < NKT - 4; ++t) KTILE(MF1);
        }
        const unsigned tm = __hip_atomic_load((unsigned*)(myws + WS_TMO), __ATOMIC_RELAXED, __HIP_MEMORY_SCOPE_AGENT);
        for (int t = NKT - 4; t < NKT; ++t) KTILE(MF1);
        if (wid == 0) STAMP(3);
        const bool bad = tm == call;
        char* ocol = p.myout + (size_t)(cb1 * BN + wc * 64 + (fq & 1) * 16 + (fq >> 1) * 8) * 2;
#pragma unroll
        for (int i = 0; i < MF1 * NF / 2; ++i) {
            const int tr = wr * (BM1 / 2) + (i >> 1) * 16 + fr;
            const int orow = tr < BROWS ? src1 * MLOC + AROWS + h1 * BROWS + tr : rank * MLOC + rb1 * XROWS + (tr - BROWS);
            u32x4 wv = finish_pair(acc[2 * i], acc[2 * i + 1]);
            if (bad) wv = (u32x4){0x7fc07fc0u, 0x7fc07fc0u, 0x7fc07fc0u, 0x7fc07fc0u};
            if (tr < BROWS + XROWS) *(u32x4*)(ocol + (size_t)orow * NOUT * 2 + (i & 1) * 64) = wv;
        }
        if (wid == 0) STAMP(4);
    } else if (wid < 6) {
        const int lw = wid - 4;
        unsigned voffA[BM0 / 16], voffB[4][2];
#pragma unroll
        for (int j = 0; j < BM0 / 16; ++j) voffA[j] = (unsigned)((16 * j + 8 * lw + (lane >> 3)) * (KDIM * 2) + (((lane & 7) ^ (lane >> 3)) * 16));
#pragma unroll
        for (int e = 0; e < 4; ++e)
#pragma unroll
            for (int cb = 0; cb < 2; ++cb) { const int r = (lane >> 2) & 7, j = lane >> 5, pos = lane & 3;
                voffB[e][cb] = (unsigned)((8 * (4 * lw + e) + r) * (NDIM * 2) + (4 * j + (pos ^ ((2 * e + (r >> 2)) & 3))) * 16 + cb * 128); }
        const char* A0 = p.X + (size_t)rb0 * BM0 * ROWB;
        const char* B0 = p.W + (size_t)(dest0 * NOUT + col0) * 2;
        const char* B1 = p.W + (size_t)(rank * NOUT + cb1 * BN) * 2;
        const unsigned lds0 = (unsigned)(size_t)smem;
        const unsigned ldA_dst = lds0 + BBYTES + lw * 1024, ldB_dst = lds0 + BN * 64 * lw;
        int ld_stage = 0;
#define LOADB(Bp, kt) const char* sB = (Bp) + (size_t)(kt) * (64 * NDIM * 2); \
      const unsigned dA = ldA_dst + ld_stage * STAGE, dB = ldB_dst + ld_stage * STAGE; ld_stage = (ld_stage + 1) & (NS - 1); \
      _Pragma("unroll") for (int e = 0; e < 4; ++e) _Pragma("unroll") for (int cb = 0; cb < 2; ++cb) glds(voffB[e][cb], sB, dB + e * BN * 16 + cb * 1024);
#define LOADT0(kt) do { LOADB(B0, kt) const char* sA = A0 + (size_t)(kt) * 128; \
      _Pragma("unroll") for (int j = 0; j < BM0 / 16; ++j) glds(voffA[j], sA, dA + j * 2048); } while (0)
#define LOADT1(kt) do { LOADB(B1, kt) const char* sA = A1 + (size_t)(kt) * 128; const char* sX = X1 + (size_t)(kt) * 128; \
      _Pragma("unroll") for (int j = 0; j < BROWS / 16; ++j) glds(voffA[j], sA, dA + j * 2048); \
      glds(voffX[0], sX, dA + 6 * 2048); glds(voffX[1], sX, dA + 7 * 2048); } while (0)
        unsigned voffX[2];
#pragma unroll
        for (int j = 0; j < 2; ++j) { const int r = 16 * j + 8 * lw + (lane >> 3); voffX[j] = (unsigned)((r < XROWS ? r : XROWS - 1) * (KDIM * 2) + (((lane & 7) ^ (lane >> 3)) * 16)); }
        const char* X1 = p.X + (size_t)rb1 * XROWS * ROWB;
        if (!isC) {
#pragma unroll
            for (int t = 0; t < NS; ++t) LOADT0(t);
        }
        PIN_CALL();
        const char* A1 = (src1 == rank) ? p.X + (size_t)(AROWS + h1 * BROWS) * ROWB
                                        : myws + WS_G + (size_t)(call & 1) * G_PAR + (size_t)src1 * G_SRC + (size_t)h1 * BROWS * ROWB;
        int t1 = NS;
        if (isC) {
            BAR;
#pragma unroll
            for (int t = 0; t < NS; ++t) LOADT1(t);
            wait_vm<3 * NLD1>(); BAR;
        } else {
            wait_vm<3 * NLD0>(); BAR;
            for (int t = 0; t < NKT - NS; ++t) { wait_vm<2 * NLD0>(); BAR; LOADT0(t + NS); }
            wait_vm<2 * NLD0>(); BAR; LOADT1(0);
            wait_vm<NLD0 + NLD1>(); BAR; LOADT1(1);
            t1 = 2;
        }
        for (int t = t1; t < NKT; ++t) { wait_vm<2 * NLD1>(); BAR; LOADT1(t); }
        for (int t = 0; t < NS; ++t) { wait_vm<2 * NLD1>(); BAR; LOADT1(t); }
        wait_vm<0>();
    } else if (wid == 6) {
        PIN_CALL();
        if (isC) {
            BAR; BAR;
            for (int n = 0; n < NKT; ++n) BAR;
        } else {
        const int sidx = (int)((blockIdx.x >> 3) * 3 + (blockIdx.x & 7));
        char* optr = p.out[dest0] + ((size_t)(rank * MLOC + rb0 * BM0 + fr) * NOUT + col0 + (fq & 1) * 16 + (fq >> 1) * 8) * 2;
        LAS char* xd = smem + lane * 16;
        u32x4 ob[40];
        BAR;
        int n = 0;
#define UNTIL(t_) do { const int tgt_ = (t_) < 2 * NKT ? (t_) : 2 * NKT; while (n < tgt_) { BAR; ++n; } } while (0)
#define CVT(j) do { constexpr int mf_ = (j) / 8, w_ = ((j) >> 1) & 3, o_ = dump_slot(w_, mf_ & 1) + ((j) & 1) * 2048; \
            ob[j] = finish_pair(*(LAS f32x4*)(xd + o_), *(LAS f32x4*)(xd + o_ + 1024)); } while (0)
#define OB 4
#define OST(j_) store_wt(optr + (size_t)(((((j_) >> 1) & 3) >> 1) * (BM0 / 2) + ((j_) / 8) * 16) * NOUT * 2 + (((j_) >> 1) & 1) * 128 + ((j_) & 1) * 64, ob[j_])
        int nb = 0;
        const int so2 = sidx % ((OSTEP * OB + 7) / 8);
#define TRY_BURST(avail) do { \
            if ((nb + 1) * OB <= (avail) && n >= OFIRST + (nb * OSTEP * OB) / 8 + so2) { \
                _Pragma("unroll") for (int b_ = 0; b_ < (avail) / OB; ++b_) if (nb == b_) { _Pragma("unroll") for (int k_ = 0; k_ < OB; ++k_) OST(b_ * OB + k_); } \
                ++nb; } } while (0)
        UNTIL(NKT + 1);
#define CV1(j) do { CVT(j); TRY_BURST((j) + 1); UNTIL(n + 1); } while (0)
#define CV8(b) do { CV1((b) * 8 + 0); CV1((b) * 8 + 1); CV1((b) * 8 + 2); CV1((b) * 8 + 3); CV1((b) * 8 + 4); CV1((b) * 8 + 5); CV1((b) * 8 + 6); CV1((b) * 8 + 7); } while (0)
        CV8(0); CV8(1);
        while (n < NKT + DUMP2_T + 2) { TRY_BURST(16); BAR; ++n; }
        CV8(2); CV8(3);
        while (n < NKT + DUMP3_T + 2) { TRY_BURST(32); BAR; ++n; }
        CV8(4);
        while (n < 2 * NKT) { TRY_BURST(40); BAR; ++n; }
        n = 1 << 20;
        while (nb * OB < 40) TRY_BURST(40);
        wait_vm<0>();
        if (lane == 0) __hip_atomic_fetch_add((unsigned*)(p.ws[dest0] + WS_OCNT + (size_t)rank * 128), 1u, __ATOMIC_RELAXED, __HIP_MEMORY_SCOPE_SYSTEM);
        }
        STAMP(5);
        if (lane == 0) {
            const unsigned tk = __hip_atomic_fetch_add((unsigned*)(myws + WS_DONE), 1u, __ATOMIC_RELAXED, __HIP_MEMORY_SCOPE_AGENT);
            if ((tk + 1u) % NWG == 0u) __hip_atomic_store((unsigned*)(myws + WS_CALL), call, __ATOMIC_RELAXED, __HIP_MEMORY_SCOPE_AGENT);
        }
    } else {
        PIN_CALL();
        unsigned* tmo = (unsigned*)(myws + WS_TMO);
        const unsigned* gc = (const unsigned*)(myws + WS_GCNT + (size_t)src1 * NCHUNK * 128);
        int n = 0, ch0 = 0;
        if (isC) {
            if (src1 != rank) spin_ge(gc, NWG * call, tmo, call);
            asm volatile("" ::: "memory");
            BAR;
            n = NKT; ch0 = 1;
        }
        BAR;
        for (int ch = ch0; ch < NCHUNK; ++ch) {
            const int need = NKT - NS + (NKT / NCHUNK) * ch;
            while (n < need - 6) { BAR; ++n; }
            if (src1 != rank) {
                unsigned v = __hip_atomic_load(gc + ch * 32, __ATOMIC_RELAXED, __HIP_MEMORY_SCOPE_SYSTEM);
                BAR; BAR; BAR; BAR; BAR; n += 5;
                if ((unsigned)__builtin_amdgcn_readfirstlane(v) < NWG * call) spin_ge(gc + ch * 32, NWG * call, tmo, call);
                asm volatile("" ::: "memory");
            }
        }
        while (n < 2 * NKT) { BAR; ++n; }
        if (blockIdx.x < NRANK && (int)blockIdx.x != rank) {
            spin_ge((const unsigned*)(myws + WS_OCNT + (size_t)blockIdx.x * 128), 32u * call, tmo, call);
            __builtin_amdgcn_fence(__ATOMIC_ACQUIRE, "");
            if (__hip_atomic_load(tmo, __ATOMIC_RELAXED, __HIP_MEMORY_SCOPE_AGENT) == call && lane == 0)
                *(unsigned short*)(p.myout + (size_t)(rank * MLOC) * NOUT * 2) = 0x7fc0;
        }
        STAMP(6);
    }
}

}

extern "C" void kernel_launch(int rank, int nranks, ncclComm_t comm,
                              hipStream_t stream,
                              void* const* d_in, const int* in_sizes, int n_in,
                              void* d_out, int out_size,
                              const int* in_dtypes, int out_dtype,
                              void* const* d_out_all,
                              void* const* d_ws_all, size_t ws_bytes) {
    (void)comm; (void)in_sizes; (void)n_in; (void)d_out; (void)out_size; (void)in_dtypes; (void)out_dtype;
    if (nranks != NRANK || ws_bytes < WS_NEED) { fprintf(stderr, "unsupported configuration\n"); return; }
    HIP_CHECK(hipFuncSetAttribute((const void*)gemm_a2a_kernel, hipFuncAttributeMaxDynamicSharedMemorySize, LDS_BYTES));
    Params p;
    p.X = (const char*)d_in[0];
    p.W = (const char*)d_in[1];
    for (int i = 0; i < NRANK; ++i) { p.out[i] = (char*)d_out_all[i]; p.ws[i] = (char*)d_ws_all[i]; }
    p.myws = p.ws[rank]; p.myout = p.out[rank];
    p.rank = rank;
    gemm_a2a_kernel<<<NWG, 512, LDS_BYTES, stream>>>(p);
    HIP_CHECK(hipGetLastError());
}
```

```cpp
#include <hip/hip_runtime.h>
#include <rccl/rccl.h>
#include <cstdio>

#define HIP_CHECK(expr) do { hipError_t _e = (expr); if (_e != hipSuccess) { \
    fprintf(stderr, "HIP error %s at %s:%d\n", hipGetErrorString(_e), __FILE__, __LINE__); \
    return; } } while (0)

#define LAS __attribute__((address_space(3)))
typedef short s16x4 __attribute__((ext_vector_type(4)));
typedef short s16x8 __attribute__((ext_vector_type(8)));
typedef float f32x4 __attribute__((ext_vector_type(4)));
typedef unsigned u32x4 __attribute__((ext_vector_type(4)));
typedef __bf16 bf16x8 __attribute__((ext_vector_type(8)));

namespace {

constexpr int NRANK = 8;
constexpr int MLOC = 512, KDIM = 4096, NDIM = 16384, NOUT = 2048, NKT = KDIM / 64;
constexpr int BN = 128, NF = 4, NS = 4;
constexpr int MF0 = 5, MF1 = 3, BM0 = 32 * MF0, BM1 = 32 * MF1;
constexpr int AROWS = 2 * BM0, GROWS = 2 * BM1;
constexpr int BBYTES = BN * 128, STAGE = BBYTES + BM0 * 128;
constexpr int DOFF = NS * STAGE;
constexpr int TAIL = BBYTES + BM1 * 128;
constexpr int LDS_BYTES = DOFF + 16384;
constexpr int NLD0 = 8 + BM0 / 16, NLD1 = 8 + BM1 / 16;
constexpr int NWG = 256;
constexpr int NCHUNK = 2;
constexpr size_t ROWB = (size_t)KDIM * 2;
constexpr size_t WS_CALL = 0, WS_TMO = 128, WS_DONE = 256, WS_GCNT = 1024, WS_OCNT = 8192, WS_G = 1u << 20;
constexpr size_t G_SRC = (size_t)GROWS * ROWB;
constexpr size_t G_PAR = NRANK * G_SRC;
constexpr size_t WS_NEED = WS_G + 2 * G_PAR;
constexpr unsigned long long DEADLINE_TICKS = 50000000ull;
#define GSTEP_ 11
#define OSTEP_ 12
#define OFIRST_ 68
constexpr int GSTEP = GSTEP_, OSTEP = OSTEP_, OFIRST = OFIRST_;
constexpr int DUMP2_T = 28;

struct Params { const char* X; const char* W; char* out[NRANK]; char* ws[NRANK]; char* myws; char* myout; int rank; };

template <int N> __device__ __forceinline__ void wait_vm() { asm volatile("s_waitcnt vmcnt(%0)" :: "n"(N) : "memory"); }
__device__ __forceinline__ void wait_lgkm0() { asm volatile("s_waitcnt lgkmcnt(0)" ::: "memory"); }
__device__ __forceinline__ unsigned cvt_pk_bf16(float lo, float hi) { unsigned r; asm("v_cvt_pk_bf16_f32 %0, %1, %2" : "=v"(r) : "v"(lo), "v"(hi)); return r; }
__device__ __forceinline__ float silu(float y) {
    float c = __builtin_amdgcn_fmed3f(y, -60.f, 60.f);
    float e = __builtin_amdgcn_exp2f(c * -1.4426950408889634f);
    return y * __builtin_amdgcn_rcpf(1.0f + e);
}
__device__ __forceinline__ void glds(unsigned voff, const char* sbase, unsigned ldsdst) {
    asm volatile("s_mov_b32 m0, %2\n\ts_nop 0\n\tglobal_load_lds_dwordx4 %0, %1" :: "v"(voff), "s"(sbase), "s"(ldsdst) : "memory");
}
__device__ __forceinline__ void store_wt(char* p, u32x4 v) {
    asm volatile("global_store_dwordx4 %0, %1, off sc0 sc1\n\ts_nop 1" :: "v"(p), "v"(v) : "memory");
}
__device__ __forceinline__ u32x4 finish_pair(f32x4 v0, f32x4 v1) {
    unsigned ax = cvt_pk_bf16(silu(v0[0]), silu(v0[1])), ay = cvt_pk_bf16(silu(v0[2]), silu(v0[3]));
    unsigned bx = cvt_pk_bf16(silu(v1[0]), silu(v1[1])), by = cvt_pk_bf16(silu(v1[2]), silu(v1[3]));
    auto rx = __builtin_amdgcn_permlane16_swap(ax, bx, false, false);
    auto ry = __builtin_amdgcn_permlane16_swap(ay, by, false, false);
    return (u32x4){rx[0], ry[0], rx[1], ry[1]};
}
__device__ __forceinline__ constexpr int dump_slot(int w, int s) { return s == 0 ? DOFF + w * 4096 : w * STAGE + TAIL + (s - 1) * 4096; }
#define STAMP(i) do { } while (0)
#define BAR __builtin_amdgcn_s_barrier()
#define SCHED __builtin_amdgcn_sched_barrier(0)

__device__ __forceinline__ void spin_ge(const unsigned* flag, unsigned want, unsigned* tmo, unsigned call) {
    const unsigned long long t0 = __builtin_amdgcn_s_memrealtime();
    while ((unsigned)__builtin_amdgcn_readfirstlane(__hip_atomic_load(flag, __ATOMIC_RELAXED, __HIP_MEMORY_SCOPE_SYSTEM)) < want) {
        if ((unsigned)__builtin_amdgcn_readfirstlane(__hip_atomic_load(tmo, __ATOMIC_RELAXED, __HIP_MEMORY_SCOPE_AGENT)) == call
            || __builtin_amdgcn_s_memrealtime() - t0 > DEADLINE_TICKS) {
            __hip_atomic_store(tmo, call, __ATOMIC_RELAXED, __HIP_MEMORY_SCOPE_AGENT);
            break;
        }
        __builtin_amdgcn_s_sleep(2);
    }
}

__global__ __launch_bounds__(512, 2) void gemm_a2a_kernel(Params p) {
    extern __shared__ __attribute__((aligned(16))) char smem_[];
    LAS char* smem = (LAS char*)smem_;
    const int tid = threadIdx.x, wid = __builtin_amdgcn_readfirstlane(tid >> 6), lane = tid & 63;
    const int fr = lane & 15, fq = lane >> 4;
    const int rank = p.rank;
    const int xcd = blockIdx.x & 7, c = blockIdx.x >> 3;
    const int rb0 = c & 1, slot0 = c >> 1;
    const int dest0 = (rank + 1 + (slot0 & 7)) & 7;
    const int col0 = (2 * xcd + (slot0 >> 3)) * BN;
    const int rb1 = 4 * (xcd & 3) + (c & 3), cb1 = 8 * (xcd >> 2) + (c >> 2);
    const int src1 = rb1 >> 1, h1 = rb1 & 1;
    char* const myws = p.myws;
    unsigned call;
#define PIN_CALL() do { SCHED; call = *(const __attribute__((address_space(4))) unsigned*)(size_t)(myws + WS_CALL) + 1u; asm volatile("" : "+s"(call)); } while (0)

    if (wid < 4) {
        const int wr = wid >> 1, wc = wid & 1;
        __builtin_amdgcn_s_setprio(3);
        const unsigned asw = (unsigned)(BBYTES + fr * 128 + ((fq ^ (fr & 7)) * 16));
        unsigned aoff0 = asw + wr * (BM0 / 2) * 128, aoff1 = aoff0 ^ 64u;
        unsigned boff[2][2];
        { const int g = fq, q = fr >> 2, pp = fr & 3;
#pragma unroll
          for (int c1 = 0; c1 < 2; ++c1)
#pragma unroll
            for (int t = 0; t < 2; ++t)
              boff[c1][t] = (unsigned)(BN * 16 * g + 1024 * wc + 64 * (4 * t + q) + 16 * ((2 * c1 + (pp >> 1)) ^ ((2 * g + t) & 3)) + 8 * (pp & 1)); }
        f32x4 acc[MF0 * NF], oldv[2 * NF];
#pragma unroll
        for (int i = 0; i < MF0 * NF; ++i) acc[i] = (f32x4){0.f, 0.f, 0.f, 0.f};
        s16x8 aA[MF0], bA[NF], aB[MF0], bB[NF];
#define RD(MF_, a_, b_, stg, ks) do { LAS char* sb_ = smem + (stg) * STAGE; \
      _Pragma("unroll") for (int nf = 0; nf < NF; ++nf) { \
        s16x4 lo = __builtin_amdgcn_ds_read_tr16_b64_v4i16((LAS s16x4*)(sb_ + boff[nf & 1][0] + (ks) * (BN * 64) + (nf >> 1) * 512)); \
        s16x4 hi = __builtin_amdgcn_ds_read_tr16_b64_v4i16((LAS s16x4*)(sb_ + boff[nf & 1][1] + (ks) * (BN * 64) + (nf >> 1) * 512)); \
        b_[nf] = __builtin_shufflevector(lo, hi, 0, 1, 2, 3, 4, 5, 6, 7); } \
      _Pragma("unroll") for (int mf = 0; mf < (MF_); ++mf) a_[mf] = *(LAS s16x8*)(sb_ + ((ks) ? aoff1 : aoff0) + mf * 2048); } while (0)
#define MM(MF_, a_, b_) do { _Pragma("unroll") for (int mf = 0; mf < (MF_); ++mf) _Pragma("unroll") for (int nf = 0; nf < NF; ++nf) \
        acc[mf * NF + nf] = __builtin_amdgcn_mfma_f32_16x16x32_bf16(__builtin_bit_cast(bf16x8, b_[nf]), __builtin_bit_cast(bf16x8, a_[mf]), acc[mf * NF + nf], 0, 0, 0); } while (0)
#define ILV(NM_, NR_) do { _Pragma("unroll") for (int i_ = 0; i_ < (NR_); ++i_) { __builtin_amdgcn_sched_group_barrier(0x008, 1, 0); __builtin_amdgcn_sched_group_barrier(0x100, 1, 0); } \
      __builtin_amdgcn_sched_group_barrier(0x008, (NM_) - (NR_), 0); } while (0)
#define KTILE2(MF_, MFN_, PRE) do { SCHED; RD(MF_, aB, bB, st, 1); MM(MF_, aA, bA); ILV((MF_) * NF, (MF_) + 2 * NF); SCHED; wait_lgkm0(); BAR; SCHED; \
      st = (st + 1) & (NS - 1); PRE; RD(MFN_, aA, bA, st, 0); MM(MF_, aB, bB); ILV((MF_) * NF, (MFN_) + 2 * NF); SCHED; } while (0)
#define KTILE(MF_) KTILE2(MF_, MF_, )
        if (wid == 0) STAMP(0);
        PIN_CALL();
        BAR;
        if (wid == 0) STAMP(1);
        int st = 0;
        RD(MF0, aA, bA, 0, 0);
        for (int t = 0; t < NKT - 1; ++t) KTILE(MF0);
        KTILE2(MF0, MF1, aoff0 = asw + wr * (BM1 / 2) * 128; aoff1 = aoff0 ^ 64u);
        if (wid == 0) STAMP(2);
        {
            LAS char* d0 = smem + DOFF + wid * 4096 + lane * 16;
            LAS char* d1 = smem + wid * STAGE + TAIL + lane * 16;
#pragma unroll
            for (int i = 0; i < NF; ++i) *(LAS f32x4*)(d0 + i * 1024) = acc[i];
#pragma unroll
            for (int i = 0; i < 2 * NF; ++i) { *(LAS f32x4*)(d1 + i * 1024) = acc[NF + i]; oldv[i] = acc[3 * NF + i]; }
#pragma unroll
            for (int i = 0; i < MF1 * NF; ++i) acc[i] = (f32x4){0.f, 0.f, 0.f, 0.f};
            for (int t = 0; t <= DUMP2_T; ++t) KTILE(MF1);
#pragma unroll
            for (int i = 0; i < 2 * NF; ++i) *(LAS f32x4*)(d1 + i * 1024) = oldv[i];
        }
        for (int t = DUMP2_T + 1; t < NKT - 4; ++t) KTILE(MF1);
        const unsigned tm = __hip_atomic_load((unsigned*)(myws + WS_TMO), __ATOMIC_RELAXED, __HIP_MEMORY_SCOPE_AGENT);
        for (int t = NKT - 4; t < NKT; ++t) KTILE(MF1);
        if (wid == 0) STAMP(3);
        const bool bad = tm == call;
        char* optr = p.myout + ((size_t)(src1 * MLOC + AROWS + h1 * BM1 + wr * (BM1 / 2) + fr) * NOUT + cb1 * BN + wc * 64 + (fq & 1) * 16 + (fq >> 1) * 8) * 2;
#pragma unroll
        for (int i = 0; i < MF1 * NF / 2; ++i) {
            u32x4 wv = finish_pair(acc[2 * i], acc[2 * i + 1]);
            if (bad) wv = (u32x4){0x7fc07fc0u, 0x7fc07fc0u, 0x7fc07fc0u, 0x7fc07fc0u};
            *(u32x4*)(optr + (size_t)(i >> 1) * 16 * NOUT * 2 + (i & 1) * 64) = wv;
        }
        if (wid == 0) STAMP(4);
    } else if (wid < 6) {
        const int lw = wid - 4;
        unsigned voffA[BM0 / 16], voffB[4][2];
#pragma unroll
        for (int j = 0; j < BM0 / 16; ++j) voffA[j] = (unsigned)((16 * j + 8 * lw + (lane >> 3)) * (KDIM * 2) + (((lane & 7) ^ (lane >> 3)) * 16));
#pragma unroll
        for (int e = 0; e < 4; ++e)
#pragma unroll
            for (int cb = 0; cb < 2; ++cb) { const int r = (lane >> 2) & 7, j = lane >> 5, pos = lane & 3;
                voffB[e][cb] = (unsigned)((8 * (4 * lw + e) + r) * (NDIM * 2) + (4 * j + (pos ^ ((2 * e + (r >> 2)) & 3))) * 16 + cb * 128); }
        const char* A0 = p.X + (size_t)rb0 * BM0 * ROWB;
        const char* B0 = p.W + (size_t)(dest0 * NOUT + col0) * 2;
        const char* B1 = p.W + (size_t)(rank * NOUT + cb1 * BN) * 2;
        const unsigned lds0 = (unsigned)(size_t)smem;
        const unsigned ldA_dst = lds0 + BBYTES + lw * 1024, ldB_dst = lds0 + BN * 64 * lw;
        int ld_stage = 0;
#define LOADT(BM_, Ap, Bp, kt) do { const char* sA = (Ap) + (size_t)(kt) * 128; const char* sB = (Bp) + (size_t)(kt) * (64 * NDIM * 2); \
      const unsigned dA = ldA_dst + ld_stage * STAGE, dB = ldB_dst + ld_stage * STAGE; ld_stage = (ld_stage + 1) & (NS - 1); \
      _Pragma("unroll") for (int e = 0; e < 4; ++e) _Pragma("unroll") for (int cb = 0; cb < 2; ++cb) glds(voffB[e][cb], sB, dB + e * BN * 16 + cb * 1024); \
      _Pragma("unroll") for (int j = 0; j < (BM_) / 16; ++j) glds(voffA[j], sA, dA + j * 2048); } while (0)
#pragma unroll
        for (int t = 0; t < NS; ++t) LOADT(BM0, A0, B0, t);
        PIN_CALL();
        wait_vm<3 * NLD0>(); BAR;
        const char* A1 = (src1 == rank) ? p.X + (size_t)(AROWS + h1 * BM1) * ROWB
                                        : myws + WS_G + (size_t)(call & 1) * G_PAR + (size_t)src1 * G_SRC + (size_t)h1 * BM1 * ROWB;
        for (int t = 0; t < NKT - NS; ++t) { wait_vm<2 * NLD0>(); BAR; LOADT(BM0, A0, B0, t + NS); }
        wait_vm<2 * NLD0>(); BAR; LOADT(BM1, A1, B1, 0);
        wait_vm<NLD0 + NLD1>(); BAR; LOADT(BM1, A1, B1, 1);
        for (int t = 2; t < NKT; ++t) { wait_vm<2 * NLD1>(); BAR; LOADT(BM1, A1, B1, t); }
        for (int t = 0; t < NS; ++t) { wait_vm<2 * NLD1>(); BAR; LOADT(BM1, A1, B1, t); }
        wait_vm<0>();
    } else if (wid == 6) {
        u32x4 xr[6];
        unsigned poff[3];
#pragma unroll
        for (int i = 0; i < 3; ++i) { const unsigned q = blockIdx.x * 3 + i; poff[i] = (q >> 2) * (unsigned)ROWB + (q & 3) * 1024 + lane * 16; }
#pragma unroll
        for (int i = 0; i < 6; ++i) xr[i] = *(const u32x4*)(p.X + (size_t)AROWS * ROWB + poff[i % 3] + (i / 3) * 4096);
        const unsigned pl = (unsigned)(rank + 1 + (lane < 7 ? lane : 0)) & 7u;
        char* peer_ws_l = p.ws[0];
#pragma unroll
        for (int q = 1; q < NRANK; ++q) peer_ws_l = (pl == (unsigned)q) ? p.ws[q] : peer_ws_l;
        unsigned* gcnt_l = (unsigned*)(peer_ws_l + WS_GCNT + (size_t)rank * NCHUNK * 128);
        asm volatile("" : "+v"(gcnt_l));
        PIN_CALL();
        const size_t goff = WS_G + (size_t)(call & 1) * G_PAR + (size_t)rank * G_SRC;
        const int qrot = (int)(blockIdx.x >> 3) % 7;
        const int sidx = (int)((blockIdx.x >> 3) * 3 + (blockIdx.x & 7));
        const int soff = sidx % GSTEP, soff2 = sidx % OSTEP;
        char* optr = p.out[dest0] + ((size_t)(rank * MLOC + rb0 * BM0 + fr) * NOUT + col0 + (fq & 1) * 16 + (fq >> 1) * 8) * 2;
        LAS char* xd = smem + lane * 16;
        u32x4 ob[40];
        BAR;
        int n = 0;
#define UNTIL(t_) do { const int tgt_ = (t_) < 2 * NKT ? (t_) : 2 * NKT; while (n < tgt_) { BAR; ++n; } } while (0)
#define PUBLISH_G(ch) do { wait_vm<0>(); if (lane < 7) { unsigned* a_ = gcnt_l + (ch) * 32; asm volatile("global_atomic_add %0, %1, off sc1" :: "v"(a_), "v"(1u) : "memory"); } } while (0)
#define GPIECE(i) do { for (int q = 0; q < 7; ++q) store_wt(p.ws[(rank + 1 + (q + qrot) % 7) & 7] + goff + poff[(i) % 3] + ((i) / 3) * 4096, xr[i]); } while (0)
#define CVT(j) do { constexpr int mf_ = (j) / 8, w_ = ((j) >> 1) & 3, o_ = dump_slot(w_, mf_ < 3 ? mf_ : mf_ - 2) + ((j) & 1) * 2048; \
            ob[j] = finish_pair(*(LAS f32x4*)(xd + o_), *(LAS f32x4*)(xd + o_ + 1024)); } while (0)
#define OB 4
#define OST(j_) store_wt(optr + (size_t)(((((j_) >> 1) & 3) >> 1) * (BM0 / 2) + ((j_) / 8) * 16) * NOUT * 2 + (((j_) >> 1) & 1) * 128 + ((j_) & 1) * 64, ob[j_])
        int nb = 0;
        bool gpub = false;
        const int so2 = sidx % ((OSTEP * OB + 7) / 8);
#define TRY_BURST(avail) do { \
            if (!gpub && n >= soff + 5 * GSTEP + 8) { PUBLISH_G(1); gpub = true; } \
            if ((nb + 1) * OB <= (avail) && n >= OFIRST + (nb * OSTEP * OB) / 8 + so2) { \
                _Pragma("unroll") for (int b_ = 0; b_ < (avail) / OB; ++b_) if (nb == b_) { _Pragma("unroll") for (int k_ = 0; k_ < OB; ++k_) OST(b_ * OB + k_); } \
                ++nb; } } while (0)
        wait_vm<0>();
#pragma unroll
        for (int i = 0; i < 6; ++i) asm volatile("" : "+v"(xr[i]));
#pragma unroll
        for (int i = 0; i < 6; ++i) {
            UNTIL(soff + i * GSTEP);
            if (i == 3) PUBLISH_G(0);
            GPIECE(i);
        }
        UNTIL(NKT + 1);
#define CV1(j) do { CVT(j); TRY_BURST((j) + 1); UNTIL(n + 1); } while (0)
#define CV8(b) do { CV1((b) * 8 + 0); CV1((b) * 8 + 1); CV1((b) * 8 + 2); CV1((b) * 8 + 3); CV1((b) * 8 + 4); CV1((b) * 8 + 5); CV1((b) * 8 + 6); CV1((b) * 8 + 7); } while (0)
        CV8(0); CV8(1); CV8(2);
        while (n < NKT + DUMP2_T + 2) { TRY_BURST(24); BAR; ++n; }
        CV8(3); CV8(4);
        while (n < 2 * NKT) { TRY_BURST(40); BAR; ++n; }
        n = 1 << 20;
        while (nb * OB < 40 || !gpub) TRY_BURST(40);
        wait_vm<0>();
        STAMP(5);
        if (lane == 0) {
            if (dest0 != rank)
                __hip_atomic_fetch_add((unsigned*)(p.ws[dest0] + WS_OCNT + (size_t)rank * 128), 1u, __ATOMIC_RELAXED, __HIP_MEMORY_SCOPE_SYSTEM);
            const unsigned tk = __hip_atomic_fetch_add((unsigned*)(myws + WS_DONE), 1u, __ATOMIC_RELAXED, __HIP_MEMORY_SCOPE_AGENT);
            if ((tk + 1u) % NWG == 0u) __hip_atomic_store((unsigned*)(myws + WS_CALL), call, __ATOMIC_RELAXED, __HIP_MEMORY_SCOPE_AGENT);
        }
    } else {
        PIN_CALL();
        BAR;
        unsigned* tmo = (unsigned*)(myws + WS_TMO);
        const unsigned* gc = (const unsigned*)(myws + WS_GCNT + (size_t)src1 * NCHUNK * 128);
        int n = 0;
        {
            const int tn = 6 + (int)((blockIdx.x >> 3) + (blockIdx.x & 7) * 4) % 32;
            while (n < tn) { BAR; ++n; }
            if (dest0 != rank) {
                const char* tb = p.out[dest0] + ((size_t)(rank * MLOC + rb0 * BM0) * NOUT + col0) * 2;
                const unsigned tl = (unsigned)(size_t)smem + DOFF;
#pragma unroll
                for (int i = 0; i < 3; ++i) { const int r = i * 64 + lane;
                    asm volatile("s_mov_b32 m0, %2\n\ts_nop 0\n\tglobal_load_lds_dword %0, %1" :: "v"((unsigned)((r < BM0 ? r : BM0 - 1) * NOUT * 2)), "s"(tb), "s"(tl) : "memory"); }
            }
        }
        for (int ch = 0; ch < NCHUNK; ++ch) {
            const int need = NKT - NS + (NKT / NCHUNK) * ch;
            while (n < need - 6) { BAR; ++n; }
            wait_vm<0>();
            if (src1 != rank) {
                unsigned v = __hip_atomic_load(gc + ch * 32, __ATOMIC_RELAXED, __HIP_MEMORY_SCOPE_SYSTEM);
                BAR; BAR; BAR; BAR; BAR; n += 5;
                if ((unsigned)__builtin_amdgcn_readfirstlane(v) < NWG * call) spin_ge(gc + ch * 32, NWG * call, tmo, call);
                asm volatile("" ::: "memory");
            }
        }
        while (n < 2 * NKT) { BAR; ++n; }
        if (blockIdx.x < NRANK && (int)blockIdx.x != rank) {
            spin_ge((const unsigned*)(myws + WS_OCNT + (size_t)blockIdx.x * 128), 32u * call, tmo, call);
            __builtin_amdgcn_fence(__ATOMIC_ACQUIRE, "");
            if (__hip_atomic_load(tmo, __ATOMIC_RELAXED, __HIP_MEMORY_SCOPE_AGENT) == call && lane == 0)
                *(unsigned short*)(p.myout + (size_t)(rank * MLOC) * NOUT * 2) = 0x7fc0;
        }
        STAMP(6);
    }
}

}

extern "C" void kernel_launch(int rank, int nranks, ncclComm_t comm,
                              hipStream_t stream,
                              void* const* d_in, const int* in_sizes, int n_in,
                              void* d_out, int out_size,
                              const int* in_dtypes, int out_dtype,
                              void* const* d_out_all,
                              void* const* d_ws_all, size_t ws_bytes) {
    (void)comm; (void)in_sizes; (void)n_in; (void)d_out; (void)out_size; (void)in_dtypes; (void)out_dtype;
    if (nranks != NRANK || ws_bytes < WS_NEED) { fprintf(stderr, "unsupported configuration\n"); return; }
    HIP_CHECK(hipFuncSetAttribute((const void*)gemm_a2a_kernel, hipFuncAttributeMaxDynamicSharedMemorySize, LDS_BYTES));
    Params p;
    p.X = (const char*)d_in[0];
    p.W = (const char*)d_in[1];
    for (int i = 0; i < NRANK; ++i) { p.out[i] = (char*)d_out_all[i]; p.ws[i] = (char*)d_ws_all[i]; }
    p.myws = p.ws[rank]; p.myout = p.out[rank];
    p.rank = rank;
    gemm_a2a_kernel<<<NWG, 512, LDS_BYTES, stream>>>(p);
    HIP_CHECK(hipGetLastError());
}
```

```cpp
#include <hip/hip_runtime.h>
#include <rccl/rccl.h>
#include <cstdio>

#define HIP_CHECK(expr) do { hipError_t _e = (expr); if (_e != hipSuccess) { \
    fprintf(stderr, "HIP error %s at %s:%d\n", hipGetErrorString(_e), __FILE__, __LINE__); \
    return; } } while (0)

#define LAS __attribute__((address_space(3)))
typedef short s16x4 __attribute__((ext_vector_type(4)));
typedef short s16x8 __attribute__((ext_vector_type(8)));
typedef float f32x4 __attribute__((ext_vector_type(4)));
typedef unsigned u32x4 __attribute__((ext_vector_type(4)));
typedef __bf16 bf16x8 __attribute__((ext_vector_type(8)));

namespace {

constexpr int NRANK = 8;
constexpr int MLOC = 512, KDIM = 4096, NDIM = 16384, NOUT = 2048, NKT = KDIM / 64;
constexpr int BM = 128, BN = 128, MF = 4, NF = 4, NACC = 16, NS = 4;
constexpr int ABYTES = BM * 128, BBYTES = BN * 128, STAGE = ABYTES + BBYTES;
constexpr int XOFF = NS * STAGE;
constexpr int LDS_BYTES = XOFF + 32768;
constexpr int NLD = 16;
constexpr int NWG = 256;
constexpr int NCHUNK = 4;
constexpr int GROWS = 256;
constexpr size_t ROWB = (size_t)KDIM * 2;
constexpr size_t WS_CALL = 0, WS_TMO = 128, WS_DONE = 256, WS_GCNT = 1024, WS_OCNT = 8192, WS_G = 1u << 20;
constexpr size_t G_SRC = (size_t)GROWS * ROWB;
constexpr size_t G_PAR = NRANK * G_SRC;
constexpr size_t WS_NEED = WS_G + 2 * G_PAR;
constexpr unsigned long long DEADLINE_TICKS = 50000000ull;
#define GSTEP_ 9
#define OSTEP_ 12
#define OFIRST_ 78
#define OB 4
constexpr int GSTEP = GSTEP_, OSTEP = OSTEP_, OFIRST = OFIRST_;
#define PRE_ 34
constexpr int PRE = PRE_, NGRP = (NKT - PRE) / 3, AEND = PRE + 4 * NGRP;
static_assert(PRE + 3 * NGRP == NKT, "bad interleave");
constexpr int DUMP2_T = 18;

struct Params { const char* X; const char* W; char* out[NRANK]; char* ws[NRANK]; char* myws; char* myout; int rank; };

template <int N> __device__ __forceinline__ void wait_vm() { asm volatile("s_waitcnt vmcnt(%0)" :: "n"(N) : "memory"); }
__device__ __forceinline__ void wait_lgkm0() { asm volatile("s_waitcnt lgkmcnt(0)" ::: "memory"); }
__device__ __forceinline__ unsigned cvt_pk_bf16(float lo, float hi) { unsigned r; asm("v_cvt_pk_bf16_f32 %0, %1, %2" : "=v"(r) : "v"(lo), "v"(hi)); return r; }
__device__ __forceinline__ float silu(float y) {
    float c = __builtin_amdgcn_fmed3f(y, -60.f, 60.f);
    float e = __builtin_amdgcn_exp2f(c * -1.4426950408889634f);
    return y * __builtin_amdgcn_rcpf(1.0f + e);
}
__device__ __forceinline__ void glds(unsigned voff, const char* sbase, unsigned ldsdst) {
    asm volatile("s_mov_b32 m0, %2\n\ts_nop 0\n\tglobal_load_lds_dwordx4 %0, %1" :: "v"(voff), "s"(sbase), "s"(ldsdst) : "memory");
}
__device__ __forceinline__ void store_wt(char* p, u32x4 v) {
#define WT_MOD "sc0 sc1"
    asm volatile("global_store_dwordx4 %0, %1, off " WT_MOD "\n\ts_nop 1" :: "v"(p), "v"(v) : "memory");
}
__device__ __forceinline__ u32x4 finish_pair(f32x4 v0, f32x4 v1) {
    unsigned ax = cvt_pk_bf16(silu(v0[0]), silu(v0[1])), ay = cvt_pk_bf16(silu(v0[2]), silu(v0[3]));
    unsigned bx = cvt_pk_bf16(silu(v1[0]), silu(v1[1])), by = cvt_pk_bf16(silu(v1[2]), silu(v1[3]));
    auto rx = __builtin_amdgcn_permlane16_swap(ax, bx, false, false);
    auto ry = __builtin_amdgcn_permlane16_swap(ay, by, false, false);
    return (u32x4){rx[0], ry[0], rx[1], ry[1]};
}
#define STAMP(i) do { } while (0)
#define BAR __builtin_amdgcn_s_barrier()
#define SCHED __builtin_amdgcn_sched_barrier(0)

__device__ __forceinline__ void spin_ge(const unsigned* flag, unsigned want, unsigned* tmo, unsigned call) {
    const unsigned long long t0 = __builtin_amdgcn_s_memrealtime();
    while ((unsigned)__builtin_amdgcn_readfirstlane(__hip_atomic_load(flag, __ATOMIC_RELAXED, __HIP_MEMORY_SCOPE_SYSTEM)) < want) {
        if ((unsigned)__builtin_amdgcn_readfirstlane(__hip_atomic_load(tmo, __ATOMIC_RELAXED, __HIP_MEMORY_SCOPE_AGENT)) == call
            || __builtin_amdgcn_s_memrealtime() - t0 > DEADLINE_TICKS) {
            __hip_atomic_store(tmo, call, __ATOMIC_RELAXED, __HIP_MEMORY_SCOPE_AGENT);
            break;
        }
        __builtin_amdgcn_s_sleep(2);
    }
}

__global__ __launch_bounds__(512, 2) void gemm_a2a_kernel(Params p) {
    extern __shared__ __attribute__((aligned(16))) char smem_[];
    LAS char* smem = (LAS char*)smem_;
    const int tid = threadIdx.x, wid = __builtin_amdgcn_readfirstlane(tid >> 6), lane = tid & 63;
    const int fr = lane & 15, fq = lane >> 4;
    const int rank = p.rank;
    const int xcd = blockIdx.x & 7, c = blockIdx.x >> 3;
    const int rb0 = c & 1, slot0 = c >> 1;
    const int dest0 = (rank + 1 + (slot0 & 7)) & 7;
    const int col0 = (2 * xcd + (slot0 >> 3)) * BN;
    const int rb1 = 4 * (xcd & 3) + (c & 3), cb1 = 8 * (xcd >> 2) + (c >> 2);
    const int src1 = rb1 >> 1, h1 = rb1 & 1;
    char* const myws = p.myws;
    unsigned call;
#define PIN_CALL() do { SCHED; call = *(const __attribute__((address_space(4))) unsigned*)(size_t)(myws + WS_CALL) + 1u; asm volatile("" : "+s"(call)); } while (0)

    if (wid < 4) {
        const int wr = wid >> 1, wc = wid & 1;
        __builtin_amdgcn_s_setprio(3);
        const unsigned aoff0 = (unsigned)((wr * 64 + fr) * 128 + ((fq ^ (fr & 7)) * 16));
        const unsigned aoff1 = aoff0 ^ 64u;
        unsigned boff[2][2];
        { const int g = fq, q = fr >> 2, pp = fr & 3;
#pragma unroll
          for (int c1 = 0; c1 < 2; ++c1)
#pragma unroll
            for (int t = 0; t < 2; ++t)
              boff[c1][t] = (unsigned)(ABYTES + BN * 16 * g + 1024 * wc + 64 * (4 * t + q) + 16 * ((2 * c1 + (pp >> 1)) ^ ((2 * g + t) & 3)) + 8 * (pp & 1)); }
        f32x4 acc0[NACC], acc[NACC];
#pragma unroll
        for (int i = 0; i < NACC; ++i) { acc0[i] = (f32x4){0.f, 0.f, 0.f, 0.f}; acc[i] = (f32x4){0.f, 0.f, 0.f, 0.f}; }
        s16x8 aA[MF], bA[NF], aB[MF], bB[NF];
        LAS char* xreg = smem + XOFF + wid * 8192 + lane * 16;
#define RD(a_, b_, stg, ks) do { LAS char* sb_ = smem + (stg) * STAGE; \
      _Pragma("unroll") for (int nf = 0; nf < NF; ++nf) { \
        s16x4 lo = __builtin_amdgcn_ds_read_tr16_b64_v4i16((LAS s16x4*)(sb_ + boff[nf & 1][0] + (ks) * (BN * 64) + (nf >> 1) * 512)); \
        s16x4 hi = __builtin_amdgcn_ds_read_tr16_b64_v4i16((LAS s16x4*)(sb_ + boff[nf & 1][1] + (ks) * (BN * 64) + (nf >> 1) * 512)); \
        b_[nf] = __builtin_shufflevector(lo, hi, 0, 1, 2, 3, 4, 5, 6, 7); } \
      _Pragma("unroll") for (int mf = 0; mf < MF; ++mf) a_[mf] = *(LAS s16x8*)(sb_ + ((ks) ? aoff1 : aoff0) + mf * 2048); } while (0)
#define MM(ac_, a_, b_) do { _Pragma("unroll") for (int mf = 0; mf < MF; ++mf) _Pragma("unroll") for (int nf = 0; nf < NF; ++nf) \
        ac_[mf * NF + nf] = __builtin_amdgcn_mfma_f32_16x16x32_bf16(__builtin_bit_cast(bf16x8, b_[nf]), __builtin_bit_cast(bf16x8, a_[mf]), ac_[mf * NF + nf], 0, 0, 0); } while (0)
#define ILV() do { _Pragma("unroll") for (int i_ = 0; i_ < MF + 2 * NF; ++i_) { __builtin_amdgcn_sched_group_barrier(0x008, 1, 0); __builtin_amdgcn_sched_group_barrier(0x100, 1, 0); } \
      __builtin_amdgcn_sched_group_barrier(0x008, NACC - (MF + 2 * NF), 0); } while (0)
#define KTILE() KT(acc)
#define KT(ac_) do { SCHED; RD(aB, bB, st, 1); MM(ac_, aA, bA); ILV(); SCHED; wait_lgkm0(); BAR; SCHED; \
      st = (st + 1) & (NS - 1); RD(aA, bA, st, 0); MM(ac_, aB, bB); ILV(); SCHED; } while (0)
        if (wid == 0) STAMP(0);
        PIN_CALL();
        BAR;
        if (wid == 0) STAMP(1);
        int st = 0;
        RD(aA, bA, 0, 0);
        for (int t = 0; t < PRE; ++t) KT(acc0);
        for (int g = 0; g < NGRP; ++g) { KT(acc0); KT(acc0); KT(acc0); KT(acc); }
        if (wid == 0) STAMP(2);
#pragma unroll
        for (int i = 0; i < NACC / 2; ++i) *(LAS f32x4*)(xreg + i * 1024) = acc0[i];
        for (int t = 0; t <= DUMP2_T; ++t) KTILE();
#pragma unroll
        for (int i = 0; i < NACC / 2; ++i) *(LAS f32x4*)(xreg + i * 1024) = acc0[i + NACC / 2];
        for (int t = AEND + DUMP2_T + 1; t < 2 * NKT - 4; ++t) KTILE();
        const unsigned tm = __hip_atomic_load((unsigned*)(myws + WS_TMO), __ATOMIC_RELAXED, __HIP_MEMORY_SCOPE_AGENT);
        for (int t = NKT - 4; t < NKT; ++t) KTILE();
        if (wid == 0) STAMP(3);
        const bool bad = tm == call;
        char* optr = p.myout + ((size_t)(src1 * MLOC + 256 + h1 * BM + wr * 64 + fr) * NOUT + cb1 * BN + wc * 64 + (fq & 1) * 16 + (fq >> 1) * 8) * 2;
#pragma unroll
        for (int i = 0; i < NACC / 2; ++i) {
            u32x4 wv = finish_pair(acc[2 * i], acc[2 * i + 1]);
            if (bad) wv = (u32x4){0x7fc07fc0u, 0x7fc07fc0u, 0x7fc07fc0u, 0x7fc07fc0u};
            *(u32x4*)(optr + (size_t)(i >> 1) * 16 * NOUT * 2 + (i & 1) * 64) = wv;
        }
        if (wid == 0) STAMP(4);
    } else if (wid < 6) {
        const int lw = wid - 4;
        unsigned voffA[8], voffB[4][2];
#pragma unroll
        for (int j = 0; j < 8; ++j) voffA[j] = (unsigned)((lw * 64 + j * 8 + (lane >> 3)) * (KDIM * 2) + (((lane & 7) ^ (lane >> 3)) * 16));
#pragma unroll
        for (int e = 0; e < 4; ++e)
#pragma unroll
            for (int cb = 0; cb < 2; ++cb) { const int r = (lane >> 2) & 7, j = lane >> 5, pos = lane & 3;
                voffB[e][cb] = (unsigned)((8 * (4 * lw + e) + r) * (NDIM * 2) + (4 * j + (pos ^ ((2 * e + (r >> 2)) & 3))) * 16 + cb * 128); }
        const char* A0 = p.X + (size_t)rb0 * BM * ROWB;
        const char* B0 = p.W + (size_t)(dest0 * NOUT + col0) * 2;
        const char* B1 = p.W + (size_t)(rank * NOUT + cb1 * BN) * 2;
        const unsigned lds0 = (unsigned)(size_t)smem;
        const unsigned ldA_dst = lds0 + lw * 8192, ldB_dst = lds0 + ABYTES + BN * 64 * lw;
        int ld_stage = 0;
#define LOADT(Ap, Bp, kt) do { const char* sA = (Ap) + (size_t)(kt) * 128; const char* sB = (Bp) + (size_t)(kt) * (64 * NDIM * 2); \
      const unsigned dA = ldA_dst + ld_stage * STAGE, dB = ldB_dst + ld_stage * STAGE; ld_stage = (ld_stage + 1) & (NS - 1); \
      _Pragma("unroll") for (int e = 0; e < 4; ++e) _Pragma("unroll") for (int cb = 0; cb < 2; ++cb) glds(voffB[e][cb], sB, dB + e * BN * 16 + cb * 1024); \
      _Pragma("unroll") for (int j = 0; j < 8; ++j) glds(voffA[j], sA, dA + j * 1024); } while (0)
#pragma unroll
        for (int t = 0; t < NS; ++t) LOADT(A0, B0, t);
        PIN_CALL();
        wait_vm<(NS - 1) * NLD>(); BAR;
        const char* A1 = (src1 == rank) ? p.X + (size_t)(256 + h1 * BM) * ROWB
                                        : myws + WS_G + (size_t)(call & 1) * G_PAR + (size_t)src1 * G_SRC + (size_t)h1 * BM * ROWB;
        {
            int ka = NS, kb = 0;
            for (int sidx = NS; sidx < 2 * NKT; ++sidx) {
                const bool isB = sidx >= AEND || (sidx >= PRE && ((sidx - PRE) & 3) == 3);
                const char* Ap = isB ? A1 : A0; const char* Bp = isB ? B1 : B0; const int kt = isB ? kb : ka;
                kb += isB ? 1 : 0; ka += isB ? 0 : 1;
                wait_vm<(NS - 2) * NLD>(); BAR; LOADT(Ap, Bp, kt);
            }
        }
        for (int t = 0; t < NS; ++t) { wait_vm<(NS - 2) * NLD>(); BAR; LOADT(A1, B1, t); }
        wait_vm<0>();
    } else if (wid == 6) {
        u32x4 xr[8];
        const char* xsrc = p.X + (size_t)(256 + blockIdx.x) * ROWB + lane * 16;
#pragma unroll
        for (int i = 0; i < 8; ++i) xr[i] = *(const u32x4*)(xsrc + i * 1024);
        const unsigned pl = (unsigned)(rank + 1 + (lane < 7 ? lane : 0)) & 7u;
        char* peer_ws_l = p.ws[0];
#pragma unroll
        for (int q = 1; q < NRANK; ++q) peer_ws_l = (pl == (unsigned)q) ? p.ws[q] : peer_ws_l;
        unsigned* gcnt_l = (unsigned*)(peer_ws_l + WS_GCNT + (size_t)rank * NCHUNK * 128);
        asm volatile("" : "+v"(gcnt_l));
        PIN_CALL();
        const size_t goff = WS_G + (size_t)(call & 1) * G_PAR + (size_t)rank * G_SRC + (size_t)blockIdx.x * ROWB + lane * 16;
        const int qrot = (int)(blockIdx.x >> 3) % 7;
        const int soff = (int)((blockIdx.x >> 3) * 3 + (blockIdx.x & 7)) % GSTEP;
        char* optr = p.out[dest0] + ((size_t)(rank * MLOC + rb0 * BM + fr) * NOUT + col0 + (fq & 1) * 16 + (fq >> 1) * 8) * 2;
        LAS char* xd = smem + XOFF + lane * 16;
        u32x4 ob[32];
        BAR;
        int n = 0;
#define UNTIL(t_) do { const int tgt_ = (t_) < 2 * NKT ? (t_) : 2 * NKT; while (n < tgt_) { BAR; ++n; } } while (0)
#define PUBLISH_G(ch) do { wait_vm<0>(); if (lane < 7) { unsigned* a_ = gcnt_l + (ch) * 32; asm volatile("global_atomic_add %0, %1, off sc1" :: "v"(a_), "v"(1u) : "memory"); } } while (0)
#define GPIECE(i) do { for (int q = 0; q < 7; ++q) store_wt(p.ws[(rank + 1 + (q + qrot) % 7) & 7] + goff + (i) * 1024, xr[i]); } while (0)
#define CVT(j) do { constexpr int o_ = (((j) >> 2) & 3) * 8192 + (2 * ((j) & 3)) * 1024; ob[j] = finish_pair(*(LAS f32x4*)(xd + o_), *(LAS f32x4*)(xd + o_ + 1024)); } while (0)
#define OST(j_) store_wt(optr + (size_t)(((((j_) >> 2) & 3) >> 1) * 64 + ((((j_) >> 4) * 4 + ((j_) & 3)) >> 1) * 16) * NOUT * 2 + (((j_) >> 2) & 1) * 128 + ((j_) & 1) * 64, ob[j_])
        int nb = 0, gst = 0, gn = 0;
        const int so2 = (int)((blockIdx.x >> 3) * 3 + (blockIdx.x & 7)) % ((OSTEP * OB + 7) / 8);
#define TRY_JOB(avail) do { \
            if (gst == 0) { if (n >= soff + 6 * GSTEP) { PUBLISH_G(2); GPIECE(6); gst = 1; } } \
            else if (gst == 1) { if (n >= soff + 7 * GSTEP) { GPIECE(7); gst = 2; gn = n; } } \
            else if (gst == 2) { if (n >= gn + 6) { PUBLISH_G(3); gst = 3; } } \
            else if ((nb + 1) * OB <= (avail) && n >= OFIRST + (nb * OSTEP * OB) / 8 + so2) { \
                _Pragma("unroll") for (int b_ = 0; b_ < (avail) / OB; ++b_) if (nb == b_) { _Pragma("unroll") for (int k_ = 0; k_ < OB; ++k_) OST(b_ * OB + k_); } \
                ++nb; } } while (0)
        wait_vm<0>();
#pragma unroll
        for (int i = 0; i < 8; ++i) asm volatile("" : "+v"(xr[i]));
#pragma unroll
        for (int i = 0; i < 6; ++i) {
            UNTIL(soff + i * GSTEP);
            if (i == 4) PUBLISH_G(1);
            GPIECE(i);
            if (i == 1) { UNTIL(n + 6); PUBLISH_G(0); }
        }
        while (n < AEND + 1) { TRY_JOB(0); BAR; ++n; }
#define CV1(j) do { CVT(j); TRY_JOB((j) + 1); UNTIL(n + 1); } while (0)
#define CV8(b) do { CV1((b) * 8 + 0); CV1((b) * 8 + 1); CV1((b) * 8 + 2); CV1((b) * 8 + 3); CV1((b) * 8 + 4); CV1((b) * 8 + 5); CV1((b) * 8 + 6); CV1((b) * 8 + 7); } while (0)
        CV8(0); CV8(1);
        while (n < AEND + DUMP2_T + 2) { TRY_JOB(16); BAR; ++n; }
        CV8(2); CV8(3);
        while (n < 2 * NKT) { TRY_JOB(32); BAR; ++n; }
        n = 1 << 20;
        while (nb * OB < 32) TRY_JOB(32);
        wait_vm<0>();
        STAMP(5);
        if (lane == 0) {
            if (dest0 != rank)
                __hip_atomic_fetch_add((unsigned*)(p.ws[dest0] + WS_OCNT + (size_t)rank * 128), 1u, __ATOMIC_RELAXED, __HIP_MEMORY_SCOPE_SYSTEM);
            const unsigned tk = __hip_atomic_fetch_add((unsigned*)(myws + WS_DONE), 1u, __ATOMIC_RELAXED, __HIP_MEMORY_SCOPE_AGENT);
            if ((tk + 1u) % NWG == 0u) __hip_atomic_store((unsigned*)(myws + WS_CALL), call, __ATOMIC_RELAXED, __HIP_MEMORY_SCOPE_AGENT);
        }
    } else {
        {
            const char* xb = p.X + (size_t)blockIdx.x * 2 * ROWB;
            const unsigned tl = (unsigned)(size_t)smem + XOFF;
            asm volatile("s_mov_b32 m0, %2\n\ts_nop 0\n\tglobal_load_lds_dword %0, %1" :: "v"((unsigned)lane * 128u), "s"(xb), "s"(tl) : "memory");
            asm volatile("s_mov_b32 m0, %2\n\ts_nop 0\n\tglobal_load_lds_dword %0, %1" :: "v"((unsigned)lane * 128u + (unsigned)ROWB), "s"(xb), "s"(tl) : "memory");
        }
        PIN_CALL();
        BAR;
        unsigned* tmo = (unsigned*)(myws + WS_TMO);
        const unsigned* gc = (const unsigned*)(myws + WS_GCNT + (size_t)src1 * NCHUNK * 128);
        int n = 0;
        for (int ch = 0; ch < NCHUNK; ++ch) {
            const int need = (ch == 0 ? PRE + 3 : NKT + 16 * ch) - NS;
            while (n < need - 6) { BAR; ++n; }
            if (src1 != rank) {
                unsigned v = __hip_atomic_load(gc + ch * 32, __ATOMIC_RELAXED, __HIP_MEMORY_SCOPE_SYSTEM);
                BAR; BAR; BAR; BAR; BAR; n += 5;
                if ((unsigned)__builtin_amdgcn_readfirstlane(v) < NWG * call) spin_ge(gc + ch * 32, NWG * call, tmo, call);
                asm volatile("" ::: "memory");
            } else wait_vm<0>();
        }
        while (n < 2 * NKT) { BAR; ++n; }
        if (blockIdx.x < NRANK && (int)blockIdx.x != rank) {
            spin_ge((const unsigned*)(myws + WS_OCNT + (size_t)blockIdx.x * 128), 32u * call, tmo, call);
            __builtin_amdgcn_fence(__ATOMIC_ACQUIRE, "");
            if (__hip_atomic_load(tmo, __ATOMIC_RELAXED, __HIP_MEMORY_SCOPE_AGENT) == call && lane == 0)
                *(unsigned short*)(p.myout + (size_t)(rank * MLOC) * NOUT * 2) = 0x7fc0;
        }
        STAMP(6);
    }
}

}

extern "C" void kernel_launch(int rank, int nranks, ncclComm_t comm,
                              hipStream_t stream,
                              void* const* d_in, const int* in_sizes, int n_in,
                              void* d_out, int out_size,
                              const int* in_dtypes, int out_dtype,
                              void* const* d_out_all,
                              void* const* d_ws_all, size_t ws_bytes) {
    (void)comm; (void)in_sizes; (void)n_in; (void)d_out; (void)out_size; (void)in_dtypes; (void)out_dtype;
    if (nranks != NRANK || ws_bytes < WS_NEED) { fprintf(stderr, "unsupported configuration\n"); return; }
    HIP_CHECK(hipFuncSetAttribute((const void*)gemm_a2a_kernel, hipFuncAttributeMaxDynamicSharedMemorySize, LDS_BYTES));
    Params p;
    p.X = (const char*)d_in[0];
    p.W = (const char*)d_in[1];
    for (int i = 0; i < NRANK; ++i) { p.out[i] = (char*)d_out_all[i]; p.ws[i] = (char*)d_ws_all[i]; }
    p.myws = p.ws[rank]; p.myout = p.out[rank];
    p.rank = rank;
    gemm_a2a_kernel<<<NWG, 512, LDS_BYTES, stream>>>(p);
    HIP_CHECK(hipGetLastError());
}
```

```cpp
#include <hip/hip_runtime.h>
#include <rccl/rccl.h>
#include <cstdio>

#define HIP_CHECK(expr) do { hipError_t _e = (expr); if (_e != hipSuccess) { \
    fprintf(stderr, "HIP error %s at %s:%d\n", hipGetErrorString(_e), __FILE__, __LINE__); \
    return; } } while (0)

#define LAS __attribute__((address_space(3)))
typedef short s16x4 __attribute__((ext_vector_type(4)));
typedef short s16x8 __attribute__((ext_vector_type(8)));
typedef float f32x4 __attribute__((ext_vector_type(4)));
typedef unsigned u32x4 __attribute__((ext_vector_type(4)));
typedef __bf16 bf16x8 __attribute__((ext_vector_type(8)));

namespace {

constexpr int NRANK = 8;
constexpr int MLOC = 512, KDIM = 4096, NDIM = 16384, NOUT = 2048, NKT = KDIM / 64;
constexpr int BN = 128, NF = 4, NS = 4;
constexpr int MF0 = 5, MF1 = 3, BM0 = 32 * MF0, BM1 = 32 * MF1;
constexpr int AROWS = 2 * BM0, GROWS = 2 * BM1;
constexpr int BBYTES = BN * 128, STAGE = BBYTES + BM0 * 128;
constexpr int DOFF = NS * STAGE;
constexpr int TAIL = BBYTES + BM1 * 128;
constexpr int LDS_BYTES = DOFF + 16384;
constexpr int NLD0 = 8 + BM0 / 16, NLD1 = 8 + BM1 / 16;
constexpr int NWG = 256;
constexpr int NCHUNK = 2;
constexpr size_t ROWB = (size_t)KDIM * 2;
constexpr size_t WS_CALL = 0, WS_TMO = 128, WS_DONE = 256, WS_GCNT = 1024, WS_OCNT = 8192, WS_G = 1u << 20;
constexpr size_t G_SRC = (size_t)GROWS * ROWB;
constexpr size_t G_PAR = NRANK * G_SRC;
constexpr size_t WS_NEED = WS_G + 2 * G_PAR;
constexpr unsigned long long DEADLINE_TICKS = 50000000ull;
#define GSTEP_ 11
#define OSTEP_ 12
#define OFIRST_ 68
constexpr int GSTEP = GSTEP_, OSTEP = OSTEP_, OFIRST = OFIRST_;
constexpr int DUMP2_T = 28;

struct Params { const char* X; const char* W; char* out[NRANK]; char* ws[NRANK]; char* myws; char* myout; int rank; };

template <int N> __device__ __forceinline__ void wait_vm() { asm volatile("s_waitcnt vmcnt(%0)" :: "n"(N) : "memory"); }
__device__ __forceinline__ void wait_lgkm0() { asm volatile("s_waitcnt lgkmcnt(0)" ::: "memory"); }
__device__ __forceinline__ unsigned cvt_pk_bf16(float lo, float hi) { unsigned r; asm("v_cvt_pk_bf16_f32 %0, %1, %2" : "=v"(r) : "v"(lo), "v"(hi)); return r; }
__device__ __forceinline__ float silu(float y) {
    float c = __builtin_amdgcn_fmed3f(y, -60.f, 60.f);
    float e = __builtin_amdgcn_exp2f(c * -1.4426950408889634f);
    return y * __builtin_amdgcn_rcpf(1.0f + e);
}
__device__ __forceinline__ void glds(unsigned voff, const char* sbase, unsigned ldsdst) {
    asm volatile("s_mov_b32 m0, %2\n\ts_nop 0\n\tglobal_load_lds_dwordx4 %0, %1" :: "v"(voff), "s"(sbase), "s"(ldsdst) : "memory");
}
__device__ __forceinline__ void store_wt(char* p, u32x4 v) {
    asm volatile("global_store_dwordx4 %0, %1, off sc0 sc1\n\ts_nop 1" :: "v"(p), "v"(v) : "memory");
}
__device__ __forceinline__ u32x4 finish_pair(f32x4 v0, f32x4 v1) {
    unsigned ax = cvt_pk_bf16(silu(v0[0]), silu(v0[1])), ay = cvt_pk_bf16(silu(v0[2]), silu(v0[3]));
    unsigned bx = cvt_pk_bf16(silu(v1[0]), silu(v1[1])), by = cvt_pk_bf16(silu(v1[2]), silu(v1[3]));
    auto rx = __builtin_amdgcn_permlane16_swap(ax, bx, false, false);
    auto ry = __builtin_amdgcn_permlane16_swap(ay, by, false, false);
    return (u32x4){rx[0], ry[0], rx[1], ry[1]};
}
__device__ __forceinline__ constexpr int dump_slot(int w, int s) { return s == 0 ? DOFF + w * 4096 : w * STAGE + TAIL + (s - 1) * 4096; }
#define STAMP(i) do { } while (0)
#define BAR __builtin_amdgcn_s_barrier()
#define SCHED __builtin_amdgcn_sched_barrier(0)

__device__ __forceinline__ void spin_ge(const unsigned* flag, unsigned want, unsigned* tmo, unsigned call) {
    const unsigned long long t0 = __builtin_amdgcn_s_memrealtime();
    while ((unsigned)__builtin_amdgcn_readfirstlane(__hip_atomic_load(flag, __ATOMIC_RELAXED, __HIP_MEMORY_SCOPE_SYSTEM)) < want) {
        if ((unsigned)__builtin_amdgcn_readfirstlane(__hip_atomic_load(tmo, __ATOMIC_RELAXED, __HIP_MEMORY_SCOPE_AGENT)) == call
            || __builtin_amdgcn_s_memrealtime() - t0 > DEADLINE_TICKS) {
            __hip_atomic_store(tmo, call, __ATOMIC_RELAXED, __HIP_MEMORY_SCOPE_AGENT);
            break;
        }
        __builtin_amdgcn_s_sleep(2);
    }
}

__global__ __launch_bounds__(512, 2) void gemm_a2a_kernel(Params p) {
    extern __shared__ __attribute__((aligned(16))) char smem_[];
    LAS char* smem = (LAS char*)smem_;
    const int tid = threadIdx.x, wid = __builtin_amdgcn_readfirstlane(tid >> 6), lane = tid & 63;
    const int fr = lane & 15, fq = lane >> 4;
    const int rank = p.rank;
    const int xcd = blockIdx.x & 7, c = blockIdx.x >> 3;
    const int rb0 = c & 1, slot0 = c >> 1;
    const int dest0 = (rank + 1 + (slot0 & 7)) & 7;
    const int col0 = (2 * xcd + (slot0 >> 3)) * BN;
    const int rb1 = 4 * (xcd & 3) + (c & 3), cb1 = 8 * (xcd >> 2) + (c >> 2);
    const int src1 = rb1 >> 1, h1 = rb1 & 1;
    char* const myws = p.myws;
    unsigned call;
#define PIN_CALL() do { SCHED; call = *(const __attribute__((address_space(4))) unsigned*)(size_t)(myws + WS_CALL) + 1u; asm volatile("" : "+s"(call)); } while (0)

    if (wid < 4) {
        const int wr = wid >> 1, wc = wid & 1;
        __builtin_amdgcn_s_setprio(3);
        const unsigned asw = (unsigned)(BBYTES + fr * 128 + ((fq ^ (fr & 7)) * 16));
        unsigned aoff0 = asw + wr * (BM0 / 2) * 128, aoff1 = aoff0 ^ 64u;
        unsigned boff[2][2];
        { const int g = fq, q = fr >> 2, pp = fr & 3;
#pragma unroll
          for (int c1 = 0; c1 < 2; ++c1)
#pragma unroll
            for (int t = 0; t < 2; ++t)
              boff[c1][t] = (unsigned)(BN * 16 * g + 1024 * wc + 64 * (4 * t + q) + 16 * ((2 * c1 + (pp >> 1)) ^ ((2 * g + t) & 3)) + 8 * (pp & 1)); }
        f32x4 acc[MF0 * NF], oldv[2 * NF];
#pragma unroll
        for (int i = 0; i < MF0 * NF; ++i) acc[i] = (f32x4){0.f, 0.f, 0.f, 0.f};
        s16x8 aA[MF0], bA[NF], aB[MF0], bB[NF];
#define RD(MF_, a_, b_, stg, ks) do { LAS char* sb_ = smem + (stg) * STAGE; \
      _Pragma("unroll") for (int nf = 0; nf < NF; ++nf) { \
        s16x4 lo = __builtin_amdgcn_ds_read_tr16_b64_v4i16((LAS s16x4*)(sb_ + boff[nf & 1][0] + (ks) * (BN * 64) + (nf >> 1) * 512)); \
        s16x4 hi = __builtin_amdgcn_ds_read_tr16_b64_v4i16((LAS s16x4*)(sb_ + boff[nf & 1][1] + (ks) * (BN * 64) + (nf >> 1) * 512)); \
        b_[nf] = __builtin_shufflevector(lo, hi, 0, 1, 2, 3, 4, 5, 6, 7); } \
      _Pragma("unroll") for (int mf = 0; mf < (MF_); ++mf) a_[mf] = *(LAS s16x8*)(sb_ + ((ks) ? aoff1 : aoff0) + mf * 2048); } while (0)
#define MM(MF_, a_, b_) do { _Pragma("unroll") for (int mf = 0; mf < (MF_); ++mf) _Pragma("unroll") for (int nf = 0; nf < NF; ++nf) \
        acc[mf * NF + nf] = __builtin_amdgcn_mfma_f32_16x16x32_bf16(__builtin_bit_cast(bf16x8, b_[nf]), __builtin_bit_cast(bf16x8, a_[mf]), acc[mf * NF + nf], 0, 0, 0); } while (0)
#define ILV(NM_, NR_) do { _Pragma("unroll") for (int i_ = 0; i_ < (NR_); ++i_) { __builtin_amdgcn_sched_group_barrier(0x008, 1, 0); __builtin_amdgcn_sched_group_barrier(0x100, 1, 0); } \
      __builtin_amdgcn_sched_group_barrier(0x008, (NM_) - (NR_), 0); } while (0)
#define KTILE2(MF_, MFN_, PRE) do { SCHED; RD(MF_, aB, bB, st, 1); MM(MF_, aA, bA); ILV((MF_) * NF, (MF_) + 2 * NF); SCHED; wait_lgkm0(); BAR; SCHED; \
      st = (st + 1) & (NS - 1); PRE; RD(MFN_, aA, bA, st, 0); MM(MF_, aB, bB); ILV((MF_) * NF, (MFN_) + 2 * NF); SCHED; } while (0)
#define KTILE(MF_) KTILE2(MF_, MF_, )
        if (wid == 0) STAMP(0);
        PIN_CALL();
        BAR;
        if (wid == 0) STAMP(1);
        int st = 0;
        RD(MF0, aA, bA, 0, 0);
        for (int t = 0; t < NKT - 1; ++t) KTILE(MF0);
        KTILE2(MF0, MF1, aoff0 = asw + wr * (BM1 / 2) * 128; aoff1 = aoff0 ^ 64u);
        if (wid == 0) STAMP(2);
        {
            LAS char* d0 = smem + DOFF + wid * 4096 + lane * 16;
            LAS char* d1 = smem + wid * STAGE + TAIL + lane * 16;
#pragma unroll
            for (int i = 0; i < NF; ++i) *(LAS f32x4*)(d0 + i * 1024) = acc[i];
#pragma unroll
            for (int i = 0; i < 2 * NF; ++i) { *(LAS f32x4*)(d1 + i * 1024) = acc[NF + i]; oldv[i] = acc[3 * NF + i]; }
#pragma unroll
            for (int i = 0; i < MF1 * NF; ++i) acc[i] = (f32x4){0.f, 0.f, 0.f, 0.f};
            for (int t = 0; t <= DUMP2_T; ++t) KTILE(MF1);
#pragma unroll
            for (int i = 0; i < 2 * NF; ++i) *(LAS f32x4*)(d1 + i * 1024) = oldv[i];
        }
        for (int t = DUMP2_T + 1; t < NKT - 4; ++t) KTILE(MF1);
        const unsigned tm = __hip_atomic_load((unsigned*)(myws + WS_TMO), __ATOMIC_RELAXED, __HIP_MEMORY_SCOPE_AGENT);
        for (int t = NKT - 4; t < NKT; ++t) KTILE(MF1);
        if (wid == 0) STAMP(3);
        const bool bad = tm == call;
        char* optr = p.myout + ((size_t)(src1 * MLOC + AROWS + h1 * BM1 + wr * (BM1 / 2) + fr) * NOUT + cb1 * BN + wc * 64 + (fq & 1) * 16 + (fq >> 1) * 8) * 2;
#pragma unroll
        for (int i = 0; i < MF1 * NF / 2; ++i) {
            u32x4 wv = finish_pair(acc[2 * i], acc[2 * i + 1]);
            if (bad) wv = (u32x4){0x7fc07fc0u, 0x7fc07fc0u, 0x7fc07fc0u, 0x7fc07fc0u};
            *(u32x4*)(optr + (size_t)(i >> 1) * 16 * NOUT * 2 + (i & 1) * 64) = wv;
        }
        if (wid == 0) STAMP(4);
    } else if (wid < 6) {
        const int lw = wid - 4;
        unsigned voffA[BM0 / 16], voffB[4][2];
#pragma unroll
        for (int j = 0; j < BM0 / 16; ++j) voffA[j] = (unsigned)((16 * j + 8 * lw + (lane >> 3)) * (KDIM * 2) + (((lane & 7) ^ (lane >> 3)) * 16));
#pragma unroll
        for (int e = 0; e < 4; ++e)
#pragma unroll
            for (int cb = 0; cb < 2; ++cb) { const int r = (lane >> 2) & 7, j = lane >> 5, pos = lane & 3;
                voffB[e][cb] = (unsigned)((8 * (4 * lw + e) + r) * (NDIM * 2) + (4 * j + (pos ^ ((2 * e + (r >> 2)) & 3))) * 16 + cb * 128); }
        const char* A0 = p.X + (size_t)rb0 * BM0 * ROWB;
        const char* B0 = p.W + (size_t)(dest0 * NOUT + col0) * 2;
        const char* B1 = p.W + (size_t)(rank * NOUT + cb1 * BN) * 2;
        const unsigned lds0 = (unsigned)(size_t)smem;
        const unsigned ldA_dst = lds0 + BBYTES + lw * 1024, ldB_dst = lds0 + BN * 64 * lw;
        int ld_stage = 0;
#define LOADT(BM_, Ap, Bp, kt) do { const char* sA = (Ap) + (size_t)(kt) * 128; const char* sB = (Bp) + (size_t)(kt) * (64 * NDIM * 2); \
      const unsigned dA = ldA_dst + ld_stage * STAGE, dB = ldB_dst + ld_stage * STAGE; ld_stage = (ld_stage + 1) & (NS - 1); \
      _Pragma("unroll") for (int e = 0; e < 4; ++e) _Pragma("unroll") for (int cb = 0; cb < 2; ++cb) glds(voffB[e][cb], sB, dB + e * BN * 16 + cb * 1024); \
      _Pragma("unroll") for (int j = 0; j < (BM_) / 16; ++j) glds(voffA[j], sA, dA + j * 2048); } while (0)
#pragma unroll
        for (int t = 0; t < NS; ++t) LOADT(BM0, A0, B0, t);
        PIN_CALL();
        wait_vm<3 * NLD0>(); BAR;
        const char* A1 = (src1 == rank) ? p.X + (size_t)(AROWS + h1 * BM1) * ROWB
                                        : myws + WS_G + (size_t)(call & 1) * G_PAR + (size_t)src1 * G_SRC + (size_t)h1 * BM1 * ROWB;
        for (int t = 0; t < NKT - NS; ++t) { wait_vm<2 * NLD0>(); BAR; LOADT(BM0, A0, B0, t + NS); }
        wait_vm<2 * NLD0>(); BAR; LOADT(BM1, A1, B1, 0);
        wait_vm<NLD0 + NLD1>(); BAR; LOADT(BM1, A1, B1, 1);
        for (int t = 2; t < NKT; ++t) { wait_vm<2 * NLD1>(); BAR; LOADT(BM1, A1, B1, t); }
        for (int t = 0; t < NS; ++t) { wait_vm<2 * NLD1>(); BAR; LOADT(BM1, A1, B1, t); }
        wait_vm<0>();
    } else if (wid == 6) {
        u32x4 xr[6];
        unsigned poff[3];
#pragma unroll
        for (int i = 0; i < 3; ++i) { const unsigned q = blockIdx.x * 3 + i; poff[i] = (q >> 2) * (unsigned)ROWB + (q & 3) * 1024 + lane * 16; }
#pragma unroll
        for (int i = 0; i < 6; ++i) xr[i] = *(const u32x4*)(p.X + (size_t)AROWS * ROWB + poff[i % 3] + (i / 3) * 4096);
        const unsigned pl = (unsigned)(rank + 1 + (lane < 7 ? lane : 0)) & 7u;
        char* peer_ws_l = p.ws[0];
#pragma unroll
        for (int q = 1; q < NRANK; ++q) peer_ws_l = (pl == (unsigned)q) ? p.ws[q] : peer_ws_l;
        unsigned* gcnt_l = (unsigned*)(peer_ws_l + WS_GCNT + (size_t)rank * NCHUNK * 128);
        asm volatile("" : "+v"(gcnt_l));
        PIN_CALL();
        const size_t goff = WS_G + (size_t)(call & 1) * G_PAR + (size_t)rank * G_SRC;
        const int qrot = (int)(blockIdx.x >> 3) % 7;
        const int sidx = (int)((blockIdx.x >> 3) * 3 + (blockIdx.x & 7));
        const int soff = sidx % GSTEP, soff2 = sidx % OSTEP;
        char* optr = p.out[dest0] + ((size_t)(rank * MLOC + rb0 * BM0 + fr) * NOUT + col0 + (fq & 1) * 16 + (fq >> 1) * 8) * 2;
        LAS char* xd = smem + lane * 16;
        u32x4 ob[40];
        BAR;
        int n = 0;
#define UNTIL(t_) do { const int tgt_ = (t_) < 2 * NKT ? (t_) : 2 * NKT; while (n < tgt_) { BAR; ++n; } } while (0)
#define PUBLISH_G(ch) do { wait_vm<0>(); if (lane < 7) { unsigned* a_ = gcnt_l + (ch) * 32; asm volatile("global_atomic_add %0, %1, off sc1" :: "v"(a_), "v"(1u) : "memory"); } } while (0)
#define GPIECE(i) do { for (int q = 0; q < 7; ++q) store_wt(p.ws[(rank + 1 + (q + qrot) % 7) & 7] + goff + poff[(i) % 3] + ((i) / 3) * 4096, xr[i]); } while (0)
#define CVT(j) do { constexpr int mf_ = (j) / 8, w_ = ((j) >> 1) & 3, o_ = dump_slot(w_, mf_ < 3 ? mf_ : mf_ - 2) + ((j) & 1) * 2048; \
            ob[j] = finish_pair(*(LAS f32x4*)(xd + o_), *(LAS f32x4*)(xd + o_ + 1024)); } while (0)
#define OB 4
#define OST(j_) store_wt(optr + (size_t)(((((j_) >> 1) & 3) >> 1) * (BM0 / 2) + ((j_) / 8) * 16) * NOUT * 2 + (((j_) >> 1) & 1) * 128 + ((j_) & 1) * 64, ob[j_])
        int nb = 0;
        bool gpub = false;
        const int so2 = sidx % ((OSTEP * OB + 7) / 8);
#define TRY_BURST(avail) do { \
            if (!gpub && n >= soff + 5 * GSTEP + 8) { PUBLISH_G(1); gpub = true; } \
            if ((nb + 1) * OB <= (avail) && n >= OFIRST + (nb * OSTEP * OB) / 8 + so2) { \
                _Pragma("unroll") for (int b_ = 0; b_ < (avail) / OB; ++b_) if (nb == b_) { _Pragma("unroll") for (int k_ = 0; k_ < OB; ++k_) OST(b_ * OB + k_); } \
                ++nb; } } while (0)
        wait_vm<0>();
#pragma unroll
        for (int i = 0; i < 6; ++i) asm volatile("" : "+v"(xr[i]));
#pragma unroll
        for (int i = 0; i < 6; ++i) {
            UNTIL(soff + i * GSTEP);
            if (i == 3) PUBLISH_G(0);
            GPIECE(i);
        }
        UNTIL(NKT + 1);
#define CV1(j) do { CVT(j); TRY_BURST((j) + 1); UNTIL(n + 1); } while (0)
#define CV8(b) do { CV1((b) * 8 + 0); CV1((b) * 8 + 1); CV1((b) * 8 + 2); CV1((b) * 8 + 3); CV1((b) * 8 + 4); CV1((b) * 8 + 5); CV1((b) * 8 + 6); CV1((b) * 8 + 7); } while (0)
        CV8(0); CV8(1); CV8(2);
        while (n < NKT + DUMP2_T + 2) { TRY_BURST(24); BAR; ++n; }
        CV8(3); CV8(4);
        while (n < 2 * NKT) { TRY_BURST(40); BAR; ++n; }
        n = 1 << 20;
        while (nb * OB < 40 || !gpub) TRY_BURST(40);
        wait_vm<0>();
        STAMP(5);
        if (lane == 0) {
            if (dest0 != rank)
                __hip_atomic_fetch_add((unsigned*)(p.ws[dest0] + WS_OCNT + (size_t)rank * 128), 1u, __ATOMIC_RELAXED, __HIP_MEMORY_SCOPE_SYSTEM);
            const unsigned tk = __hip_atomic_fetch_add((unsigned*)(myws + WS_DONE), 1u, __ATOMIC_RELAXED, __HIP_MEMORY_SCOPE_AGENT);
            if ((tk + 1u) % NWG == 0u) __hip_atomic_store((unsigned*)(myws + WS_CALL), call, __ATOMIC_RELAXED, __HIP_MEMORY_SCOPE_AGENT);
        }
    } else {
        {
            const char* xb = p.X + (size_t)blockIdx.x * 2 * ROWB;
            const unsigned tl = (unsigned)(size_t)smem + DOFF;
            asm volatile("s_mov_b32 m0, %2\n\ts_nop 0\n\tglobal_load_lds_dword %0, %1" :: "v"((unsigned)lane * 128u), "s"(xb), "s"(tl) : "memory");
            asm volatile("s_mov_b32 m0, %2\n\ts_nop 0\n\tglobal_load_lds_dword %0, %1" :: "v"((unsigned)lane * 128u + (unsigned)ROWB), "s"(xb), "s"(tl) : "memory");
        }
        PIN_CALL();
        BAR;
        unsigned* tmo = (unsigned*)(myws + WS_TMO);
        const unsigned* gc = (const unsigned*)(myws + WS_GCNT + (size_t)src1 * NCHUNK * 128);
        int n = 0;
        for (int ch = 0; ch < NCHUNK; ++ch) {
            const int need = NKT - NS + (NKT / NCHUNK) * ch;
            while (n < need - 6) { BAR; ++n; }
            wait_vm<0>();
            if (src1 != rank) {
                unsigned v = __hip_atomic_load(gc + ch * 32, __ATOMIC_RELAXED, __HIP_MEMORY_SCOPE_SYSTEM);
                BAR; BAR; BAR; BAR; BAR; n += 5;
                if ((unsigned)__builtin_amdgcn_readfirstlane(v) < NWG * call) spin_ge(gc + ch * 32, NWG * call, tmo, call);
                asm volatile("" ::: "memory");
            }
        }
        while (n < 2 * NKT) { BAR; ++n; }
        if (blockIdx.x < NRANK && (int)blockIdx.x != rank) {
            spin_ge((const unsigned*)(myws + WS_OCNT + (size_t)blockIdx.x * 128), 32u * call, tmo, call);
            __builtin_amdgcn_fence(__ATOMIC_ACQUIRE, "");
            if (__hip_atomic_load(tmo, __ATOMIC_RELAXED, __HIP_MEMORY_SCOPE_AGENT) == call && lane == 0)
                *(unsigned short*)(p.myout + (size_t)(rank * MLOC) * NOUT * 2) = 0x7fc0;
        }
        STAMP(6);
    }
}

}

extern "C" void kernel_launch(int rank, int nranks, ncclComm_t comm,
                              hipStream_t stream,
                              void* const* d_in, const int* in_sizes, int n_in,
                              void* d_out, int out_size,
                              const int* in_dtypes, int out_dtype,
                              void* const* d_out_all,
                              void* const* d_ws_all, size_t ws_bytes) {
    (void)comm; (void)in_sizes; (void)n_in; (void)d_out; (void)out_size; (void)in_dtypes; (void)out_dtype;
    if (nranks != NRANK || ws_bytes < WS_NEED) { fprintf(stderr, "unsupported configuration\n"); return; }
    HIP_CHECK(hipFuncSetAttribute((const void*)gemm_a2a_kernel, hipFuncAttributeMaxDynamicSharedMemorySize, LDS_BYTES));
    Params p;
    p.X = (const char*)d_in[0];
    p.W = (const char*)d_in[1];
    for (int i = 0; i < NRANK; ++i) { p.out[i] = (char*)d_out_all[i]; p.ws[i] = (char*)d_ws_all[i]; }
    p.myws = p.ws[rank]; p.myout = p.out[rank];
    p.rank = rank;
    gemm_a2a_kernel<<<NWG, 512, LDS_BYTES, stream>>>(p);
    HIP_CHECK(hipGetLastError());
}
```

```cpp
#include <hip/hip_runtime.h>
#include <rccl/rccl.h>
#include <cstdio>

#define HIP_CHECK(expr) do { hipError_t _e = (expr); if (_e != hipSuccess) { \
    fprintf(stderr, "HIP error %s at %s:%d\n", hipGetErrorString(_e), __FILE__, __LINE__); \
    return; } } while (0)

#define LAS __attribute__((address_space(3)))
typedef short s16x4 __attribute__((ext_vector_type(4)));
typedef short s16x8 __attribute__((ext_vector_type(8)));
typedef float f32x4 __attribute__((ext_vector_type(4)));
typedef unsigned u32x4 __attribute__((ext_vector_type(4)));
typedef __bf16 bf16x8 __attribute__((ext_vector_type(8)));

namespace {

constexpr int NRANK = 8;
constexpr int MLOC = 512, KDIM = 4096, NDIM = 16384, NOUT = 2048, NKT = KDIM / 64;
constexpr int BM = 128, BN = 128, MF = 4, NF = 4, NACC = 16, NS = 4;
constexpr int ABYTES = BM * 128, BBYTES = BN * 128, STAGE = ABYTES + BBYTES;
constexpr int XOFF = NS * STAGE;
constexpr int LDS_BYTES = XOFF + 32768;
constexpr int NLD = 16;
constexpr int NWG = 256;
constexpr int NCHUNK = 4;
constexpr int GROWS = 256;
constexpr size_t ROWB = (size_t)KDIM * 2;
constexpr size_t WS_CALL = 0, WS_TMO = 128, WS_DONE = 256, WS_GCNT = 1024, WS_OCNT = 8192, WS_G = 1u << 20;
constexpr size_t G_SRC = (size_t)GROWS * ROWB;
constexpr size_t G_PAR = NRANK * G_SRC;
constexpr size_t WS_NEED = WS_G + 2 * G_PAR;
constexpr unsigned long long DEADLINE_TICKS = 50000000ull;
#define GSTEP_ 9
#define OSTEP_ 12
#define OFIRST_ 78
#define OB 4
constexpr int GSTEP = GSTEP_, OSTEP = OSTEP_, OFIRST = OFIRST_;
#define PRE_ 34
constexpr int PRE = PRE_, NGRP = (NKT - PRE) / 3, AEND = PRE + 4 * NGRP;
static_assert(PRE + 3 * NGRP == NKT, "bad interleave");
constexpr int DUMP2_T = 18;

struct Params { const char* X; const char* W; char* out[NRANK]; char* ws[NRANK]; char* myws; char* myout; int rank; };

template <int N> __device__ __forceinline__ void wait_vm() { asm volatile("s_waitcnt vmcnt(%0)" :: "n"(N) : "memory"); }
__device__ __forceinline__ void wait_lgkm0() { asm volatile("s_waitcnt lgkmcnt(0)" ::: "memory"); }
__device__ __forceinline__ unsigned cvt_pk_bf16(float lo, float hi) { unsigned r; asm("v_cvt_pk_bf16_f32 %0, %1, %2" : "=v"(r) : "v"(lo), "v"(hi)); return r; }
__device__ __forceinline__ float silu(float y) {
    float c = __builtin_amdgcn_fmed3f(y, -60.f, 60.f);
    float e = __builtin_amdgcn_exp2f(c * -1.4426950408889634f);
    return y * __builtin_amdgcn_rcpf(1.0f + e);
}
__device__ __forceinline__ void glds(unsigned voff, const char* sbase, unsigned ldsdst) {
    asm volatile("s_mov_b32 m0, %2\n\ts_nop 0\n\tglobal_load_lds_dwordx4 %0, %1" :: "v"(voff), "s"(sbase), "s"(ldsdst) : "memory");
}
__device__ __forceinline__ void store_wt(char* p, u32x4 v) {
    asm volatile("global_store_dwordx4 %0, %1, off sc0 sc1\n\ts_nop 1" :: "v"(p), "v"(v) : "memory");
}
__device__ __forceinline__ u32x4 finish_pair(f32x4 v0, f32x4 v1) {
    unsigned ax = cvt_pk_bf16(silu(v0[0]), silu(v0[1])), ay = cvt_pk_bf16(silu(v0[2]), silu(v0[3]));
    unsigned bx = cvt_pk_bf16(silu(v1[0]), silu(v1[1])), by = cvt_pk_bf16(silu(v1[2]), silu(v1[3]));
    auto rx = __builtin_amdgcn_permlane16_swap(ax, bx, false, false);
    auto ry = __builtin_amdgcn_permlane16_swap(ay, by, false, false);
    return (u32x4){rx[0], ry[0], rx[1], ry[1]};
}
#define STAMP(i) do { } while (0)
#define BAR __builtin_amdgcn_s_barrier()
#define SCHED __builtin_amdgcn_sched_barrier(0)

__device__ __forceinline__ void spin_ge(const unsigned* flag, unsigned want, unsigned* tmo, unsigned call) {
    const unsigned long long t0 = __builtin_amdgcn_s_memrealtime();
    while ((unsigned)__builtin_amdgcn_readfirstlane(__hip_atomic_load(flag, __ATOMIC_RELAXED, __HIP_MEMORY_SCOPE_SYSTEM)) < want) {
        if ((unsigned)__builtin_amdgcn_readfirstlane(__hip_atomic_load(tmo, __ATOMIC_RELAXED, __HIP_MEMORY_SCOPE_AGENT)) == call
            || __builtin_amdgcn_s_memrealtime() - t0 > DEADLINE_TICKS) {
            __hip_atomic_store(tmo, call, __ATOMIC_RELAXED, __HIP_MEMORY_SCOPE_AGENT);
            break;
        }
        __builtin_amdgcn_s_sleep(2);
    }
}

__global__ __launch_bounds__(512, 2) void gemm_a2a_kernel(Params p) {
    extern __shared__ __attribute__((aligned(16))) char smem_[];
    LAS char* smem = (LAS char*)smem_;
    const int tid = threadIdx.x, wid = __builtin_amdgcn_readfirstlane(tid >> 6), lane = tid & 63;
    const int fr = lane & 15, fq = lane >> 4;
    const int rank = p.rank;
    const int xcd = blockIdx.x & 7, c = blockIdx.x >> 3;
    const int rb0 = c & 1, slot0 = c >> 1;
    const int dest0 = (rank + 1 + (slot0 & 7)) & 7;
    const int col0 = (2 * xcd + (slot0 >> 3)) * BN;
    const int rb1 = 4 * (xcd & 3) + (c & 3), cb1 = 8 * (xcd >> 2) + (c >> 2);
    const int src1 = rb1 >> 1, h1 = rb1 & 1;
    char* const myws = p.myws;
    unsigned call;
#define PIN_CALL() do { SCHED; call = *(const __attribute__((address_space(4))) unsigned*)(size_t)(myws + WS_CALL) + 1u; asm volatile("" : "+s"(call)); } while (0)

    if (wid < 4) {
        const int wr = wid >> 1, wc = wid & 1;
        __builtin_amdgcn_s_setprio(3);
        const unsigned aoff0 = (unsigned)((wr * 64 + fr) * 128 + ((fq ^ (fr & 7)) * 16));
        const unsigned aoff1 = aoff0 ^ 64u;
        unsigned boff[2][2];
        { const int g = fq, q = fr >> 2, pp = fr & 3;
#pragma unroll
          for (int c1 = 0; c1 < 2; ++c1)
#pragma unroll
            for (int t = 0; t < 2; ++t)
              boff[c1][t] = (unsigned)(ABYTES + BN * 16 * g + 1024 * wc + 64 * (4 * t + q) + 16 * ((2 * c1 + (pp >> 1)) ^ ((2 * g + t) & 3)) + 8 * (pp & 1)); }
        f32x4 acc0[NACC], acc[NACC];
#pragma unroll
        for (int i = 0; i < NACC; ++i) { acc0[i] = (f32x4){0.f, 0.f, 0.f, 0.f}; acc[i] = (f32x4){0.f, 0.f, 0.f, 0.f}; }
        s16x8 aA[MF], bA[NF], aB[MF], bB[NF];
        LAS char* xreg = smem + XOFF + wid * 8192 + lane * 16;
#define RD(a_, b_, stg, ks) do { LAS char* sb_ = smem + (stg) * STAGE; \
      _Pragma("unroll") for (int nf = 0; nf < NF; ++nf) { \
        s16x4 lo = __builtin_amdgcn_ds_read_tr16_b64_v4i16((LAS s16x4*)(sb_ + boff[nf & 1][0] + (ks) * (BN * 64) + (nf >> 1) * 512)); \
        s16x4 hi = __builtin_amdgcn_ds_read_tr16_b64_v4i16((LAS s16x4*)(sb_ + boff[nf & 1][1] + (ks) * (BN * 64) + (nf >> 1) * 512)); \
        b_[nf] = __builtin_shufflevector(lo, hi, 0, 1, 2, 3, 4, 5, 6, 7); } \
      _Pragma("unroll") for (int mf = 0; mf < MF; ++mf) a_[mf] = *(LAS s16x8*)(sb_ + ((ks) ? aoff1 : aoff0) + mf * 2048); } while (0)
#define MM(ac_, a_, b_) do { _Pragma("unroll") for (int mf = 0; mf < MF; ++mf) _Pragma("unroll") for (int nf = 0; nf < NF; ++nf) \
        ac_[mf * NF + nf] = __builtin_amdgcn_mfma_f32_16x16x32_bf16(__builtin_bit_cast(bf16x8, b_[nf]), __builtin_bit_cast(bf16x8, a_[mf]), ac_[mf * NF + nf], 0, 0, 0); } while (0)
#define ILV() do { _Pragma("unroll") for (int i_ = 0; i_ < MF + 2 * NF; ++i_) { __builtin_amdgcn_sched_group_barrier(0x008, 1, 0); __builtin_amdgcn_sched_group_barrier(0x100, 1, 0); } \
      __builtin_amdgcn_sched_group_barrier(0x008, NACC - (MF + 2 * NF), 0); } while (0)
#define KTILE() KT(acc)
#define KT(ac_) do { SCHED; RD(aB, bB, st, 1); MM(ac_, aA, bA); ILV(); SCHED; wait_lgkm0(); BAR; SCHED; \
      st = (st + 1) & (NS - 1); RD(aA, bA, st, 0); MM(ac_, aB, bB); ILV(); SCHED; } while (0)
        if (wid == 0) STAMP(0);
        PIN_CALL();
        BAR;
        if (wid == 0) STAMP(1);
        int st = 0;
        RD(aA, bA, 0, 0);
        for (int t = 0; t < PRE; ++t) KT(acc0);
        for (int g = 0; g < NGRP; ++g) { KT(acc0); KT(acc0); KT(acc0); KT(acc); }
        if (wid == 0) STAMP(2);
#pragma unroll
        for (int i = 0; i < NACC / 2; ++i) *(LAS f32x4*)(xreg + i * 1024) = acc0[i];
        for (int t = 0; t <= DUMP2_T; ++t) KTILE();
#pragma unroll
        for (int i = 0; i < NACC / 2; ++i) *(LAS f32x4*)(xreg + i * 1024) = acc0[i + NACC / 2];
        for (int t = AEND + DUMP2_T + 1; t < 2 * NKT - 4; ++t) KTILE();
        const unsigned tm = __hip_atomic_load((unsigned*)(myws + WS_TMO), __ATOMIC_RELAXED, __HIP_MEMORY_SCOPE_AGENT);
        for (int t = NKT - 4; t < NKT; ++t) KTILE();
        if (wid == 0) STAMP(3);
        const bool bad = tm == call;
        char* optr = p.myout + ((size_t)(src1 * MLOC + 256 + h1 * BM + wr * 64 + fr) * NOUT + cb1 * BN + wc * 64 + (fq & 1) * 16 + (fq >> 1) * 8) * 2;
#pragma unroll
        for (int i = 0; i < NACC / 2; ++i) {
            u32x4 wv = finish_pair(acc[2 * i], acc[2 * i + 1]);
            if (bad) wv = (u32x4){0x7fc07fc0u, 0x7fc07fc0u, 0x7fc07fc0u, 0x7fc07fc0u};
            *(u32x4*)(optr + (size_t)(i >> 1) * 16 * NOUT * 2 + (i & 1) * 64) = wv;
        }
        if (wid == 0) STAMP(4);
    } else if (wid < 6) {
        const int lw = wid - 4;
        unsigned voffA[8], voffB[4][2];
#pragma unroll
        for (int j = 0; j < 8; ++j) voffA[j] = (unsigned)((lw * 64 + j * 8 + (lane >> 3)) * (KDIM * 2) + (((lane & 7) ^ (lane >> 3)) * 16));
#pragma unroll
        for (int e = 0; e < 4; ++e)
#pragma unroll
            for (int cb = 0; cb < 2; ++cb) { const int r = (lane >> 2) & 7, j = lane >> 5, pos = lane & 3;
                voffB[e][cb] = (unsigned)((8 * (4 * lw + e) + r) * (NDIM * 2) + (4 * j + (pos ^ ((2 * e + (r >> 2)) & 3))) * 16 + cb * 128); }
        const char* A0 = p.X + (size_t)rb0 * BM * ROWB;
        const char* B0 = p.W + (size_t)(dest0 * NOUT + col0) * 2;
        const char* B1 = p.W + (size_t)(rank * NOUT + cb1 * BN) * 2;
        const unsigned lds0 = (unsigned)(size_t)smem;
        const unsigned ldA_dst = lds0 + lw * 8192, ldB_dst = lds0 + ABYTES + BN * 64 * lw;
        int ld_stage = 0;
#define LOADT(Ap, Bp, kt) do { const char* sA = (Ap) + (size_t)(kt) * 128; const char* sB = (Bp) + (size_t)(kt) * (64 * NDIM * 2); \
      const unsigned dA = ldA_dst + ld_stage * STAGE, dB = ldB_dst + ld_stage * STAGE; ld_stage = (ld_stage + 1) & (NS - 1); \
      _Pragma("unroll") for (int e = 0; e < 4; ++e) _Pragma("unroll") for (int cb = 0; cb < 2; ++cb) glds(voffB[e][cb], sB, dB + e * BN * 16 + cb * 1024); \
      _Pragma("unroll") for (int j = 0; j < 8; ++j) glds(voffA[j], sA, dA + j * 1024); } while (0)
#pragma unroll
        for (int t = 0; t < NS; ++t) LOADT(A0, B0, t);
        PIN_CALL();
        wait_vm<(NS - 1) * NLD>(); BAR;
        const char* A1 = (src1 == rank) ? p.X + (size_t)(256 + h1 * BM) * ROWB
                                        : myws + WS_G + (size_t)(call & 1) * G_PAR + (size_t)src1 * G_SRC + (size_t)h1 * BM * ROWB;
        {
            int ka = NS, kb = 0;
            for (int sidx = NS; sidx < 2 * NKT; ++sidx) {
                const bool isB = sidx >= AEND || (sidx >= PRE && ((sidx - PRE) & 3) == 3);
                const char* Ap = isB ? A1 : A0; const char* Bp = isB ? B1 : B0; const int kt = isB ? kb : ka;
                kb += isB ? 1 : 0; ka += isB ? 0 : 1;
                wait_vm<(NS - 2) * NLD>(); BAR; LOADT(Ap, Bp, kt);
            }
        }
        for (int t = 0; t < NS; ++t) { wait_vm<(NS - 2) * NLD>(); BAR; LOADT(A1, B1, t); }
        wait_vm<0>();
    } else if (wid == 6) {
        u32x4 xr[8];
        const char* xsrc = p.X + (size_t)(256 + blockIdx.x) * ROWB + lane * 16;
#pragma unroll
        for (int i = 0; i < 8; ++i) xr[i] = *(const u32x4*)(xsrc + i * 1024);
        const unsigned pl = (unsigned)(rank + 1 + (lane < 7 ? lane : 0)) & 7u;
        char* peer_ws_l = p.ws[0];
#pragma unroll
        for (int q = 1; q < NRANK; ++q) peer_ws_l = (pl == (unsigned)q) ? p.ws[q] : peer_ws_l;
        unsigned* gcnt_l = (unsigned*)(peer_ws_l + WS_GCNT + (size_t)rank * NCHUNK * 128);
        asm volatile("" : "+v"(gcnt_l));
        PIN_CALL();
        const size_t goff = WS_G + (size_t)(call & 1) * G_PAR + (size_t)rank * G_SRC + (size_t)blockIdx.x * ROWB + lane * 16;
        const int qrot = (int)(blockIdx.x >> 3) % 7;
        const int soff = (int)((blockIdx.x >> 3) * 3 + (blockIdx.x & 7)) % GSTEP;
        char* optr = p.out[dest0] + ((size_t)(rank * MLOC + rb0 * BM + fr) * NOUT + col0 + (fq & 1) * 16 + (fq >> 1) * 8) * 2;
        LAS char* xd = smem + XOFF + lane * 16;
        u32x4 ob[32];
        BAR;
        int n = 0;
#define UNTIL(t_) do { const int tgt_ = (t_) < 2 * NKT ? (t_) : 2 * NKT; while (n < tgt_) { BAR; ++n; } } while (0)
#define PUBLISH_G(ch) do { wait_vm<0>(); if (lane < 7) { unsigned* a_ = gcnt_l + (ch) * 32; asm volatile("global_atomic_add %0, %1, off sc1" :: "v"(a_), "v"(1u) : "memory"); } } while (0)
#define GPIECE(i) do { for (int q = 0; q < 7; ++q) store_wt(p.ws[(rank + 1 + (q + qrot) % 7) & 7] + goff + (i) * 1024, xr[i]); } while (0)
#define CVT(j) do { constexpr int o_ = (((j) >> 2) & 3) * 8192 + (2 * ((j) & 3)) * 1024; ob[j] = finish_pair(*(LAS f32x4*)(xd + o_), *(LAS f32x4*)(xd + o_ + 1024)); } while (0)
#define OST(j_) store_wt(optr + (size_t)(((((j_) >> 2) & 3) >> 1) * 64 + ((((j_) >> 4) * 4 + ((j_) & 3)) >> 1) * 16) * NOUT * 2 + (((j_) >> 2) & 1) * 128 + ((j_) & 1) * 64, ob[j_])
        int nb = 0, gst = 0, gn = 0;
        const int so2 = (int)((blockIdx.x >> 3) * 3 + (blockIdx.x & 7)) % ((OSTEP * OB + 7) / 8);
#define TRY_JOB(avail) do { \
            if (gst == 0) { if (n >= soff + 6 * GSTEP) { PUBLISH_G(2); GPIECE(6); gst = 1; } } \
            else if (gst == 1) { if (n >= soff + 7 * GSTEP) { GPIECE(7); gst = 2; gn = n; } } \
            else if (gst == 2) { if (n >= gn + 6) { PUBLISH_G(3); gst = 3; } } \
            else if ((nb + 1) * OB <= (avail) && n >= OFIRST + (nb * OSTEP * OB) / 8 + so2) { \
                _Pragma("unroll") for (int b_ = 0; b_ < (avail) / OB; ++b_) if (nb == b_) { _Pragma("unroll") for (int k_ = 0; k_ < OB; ++k_) OST(b_ * OB + k_); } \
                ++nb; } } while (0)
        wait_vm<0>();
#pragma unroll
        for (int i = 0; i < 8; ++i) asm volatile("" : "+v"(xr[i]));
#pragma unroll
        for (int i = 0; i < 6; ++i) {
            UNTIL(soff + i * GSTEP);
            if (i == 4) PUBLISH_G(1);
            GPIECE(i);
            if (i == 1) { UNTIL(n + 6); PUBLISH_G(0); }
        }
        while (n < AEND + 1) { TRY_JOB(0); BAR; ++n; }
#define CV1(j) do { CVT(j); TRY_JOB((j) + 1); UNTIL(n + 1); } while (0)
#define CV8(b) do { CV1((b) * 8 + 0); CV1((b) * 8 + 1); CV1((b) * 8 + 2); CV1((b) * 8 + 3); CV1((b) * 8 + 4); CV1((b) * 8 + 5); CV1((b) * 8 + 6); CV1((b) * 8 + 7); } while (0)
        CV8(0); CV8(1);
        while (n < AEND + DUMP2_T + 2) { TRY_JOB(16); BAR; ++n; }
        CV8(2); CV8(3);
        while (n < 2 * NKT) { TRY_JOB(32); BAR; ++n; }
        n = 1 << 20;
        while (nb * OB < 32) TRY_JOB(32);
        wait_vm<0>();
        STAMP(5);
        if (lane == 0) {
            if (dest0 != rank)
                __hip_atomic_fetch_add((unsigned*)(p.ws[dest0] + WS_OCNT + (size_t)rank * 128), 1u, __ATOMIC_RELAXED, __HIP_MEMORY_SCOPE_SYSTEM);
            const unsigned tk = __hip_atomic_fetch_add((unsigned*)(myws + WS_DONE), 1u, __ATOMIC_RELAXED, __HIP_MEMORY_SCOPE_AGENT);
            if ((tk + 1u) % NWG == 0u) __hip_atomic_store((unsigned*)(myws + WS_CALL), call, __ATOMIC_RELAXED, __HIP_MEMORY_SCOPE_AGENT);
        }
    } else {
        PIN_CALL();
        BAR;
        unsigned* tmo = (unsigned*)(myws + WS_TMO);
        const unsigned* gc = (const unsigned*)(myws + WS_GCNT + (size_t)src1 * NCHUNK * 128);
        int n = 0;
        for (int ch = 0; ch < NCHUNK; ++ch) {
            const int need = (ch == 0 ? PRE + 3 : NKT + 16 * ch) - NS;
            while (n < need - 6) { BAR; ++n; }
            if (src1 != rank) {
                unsigned v = __hip_atomic_load(gc + ch * 32, __ATOMIC_RELAXED, __HIP_MEMORY_SCOPE_SYSTEM);
                BAR; BAR; BAR; BAR; BAR; n += 5;
                if ((unsigned)__builtin_amdgcn_readfirstlane(v) < NWG * call) spin_ge(gc + ch * 32, NWG * call, tmo, call);
                asm volatile("" ::: "memory");
            }
        }
        while (n < 2 * NKT) { BAR; ++n; }
        if (blockIdx.x < NRANK && (int)blockIdx.x != rank) {
            spin_ge((const unsigned*)(myws + WS_OCNT + (size_t)blockIdx.x * 128), 32u * call, tmo, call);
            __builtin_amdgcn_fence(__ATOMIC_ACQUIRE, "");
            if (__hip_atomic_load(tmo, __ATOMIC_RELAXED, __HIP_MEMORY_SCOPE_AGENT) == call && lane == 0)
                *(unsigned short*)(p.myout + (size_t)(rank * MLOC) * NOUT * 2) = 0x7fc0;
        }
        STAMP(6);
    }
}

}

extern "C" void kernel_launch(int rank, int nranks, ncclComm_t comm,
                              hipStream_t stream,
                              void* const* d_in, const int* in_sizes, int n_in,
                              void* d_out, int out_size,
                              const int* in_dtypes, int out_dtype,
                              void* const* d_out_all,
                              void* const* d_ws_all, size_t ws_bytes) {
    (void)comm; (void)in_sizes; (void)n_in; (void)d_out; (void)out_size; (void)in_dtypes; (void)out_dtype;
    if (nranks != NRANK || ws_bytes < WS_NEED) { fprintf(stderr, "unsupported configuration\n"); return; }
    HIP_CHECK(hipFuncSetAttribute((const void*)gemm_a2a_kernel, hipFuncAttributeMaxDynamicSharedMemorySize, LDS_BYTES));
    Params p;
    p.X = (const char*)d_in[0];
    p.W = (const char*)d_in[1];
    for (int i = 0; i < NRANK; ++i) { p.out[i] = (char*)d_out_all[i]; p.ws[i] = (char*)d_ws_all[i]; }
    p.myws = p.ws[rank]; p.myout = p.out[rank];
    p.rank = rank;
    gemm_a2a_kernel<<<NWG, 512, LDS_BYTES, stream>>>(p);
    HIP_CHECK(hipGetLastError());
}
```
